```python
import numpy as np

kernel_name = "gpu_ag_gemm_m4096_k4096_n16384_bf16_reluq8"


ABSMAX_RTOL = 0.02

COMPUTE_DTYPE = "bfloat16"
EPILOGUE = "relu_quant_int8"
IN_QUANT = None
OUT_QUANT = "int8"
IN_DTYPES = {"x": "bfloat16", "w_mat": "bfloat16"}
WORLD_SIZE = 8

def _to_bf16(x):
    u = np.ascontiguousarray(x, dtype=np.float32).view(np.uint32)
    return ((u + 0x7FFF + ((u >> 16) & 1)) & 0xFFFF0000).view(np.float32)

def setup_inputs():

    x = _to_bf16(np.random.randn(4096, 4096).astype(np.float32))
    w_mat = _to_bf16((np.random.randn(4096, 16384) / np.sqrt(4096)).astype(np.float32))
    return {"x": x, "w_mat": w_mat}

def _quant_dequant(y, amax):
    scale = amax / 127.0
    q = np.clip(np.round(y / scale), -127.0, 127.0)
    return (q * scale).astype(np.float32)
def _epilogue(y):
    y = np.maximum(y, 0.0)
    return _quant_dequant(y, np.abs(y).max())

def reference(x, w_mat):
    y = (x @ w_mat).astype(np.float32)
    return _epilogue(y)
```

```cpp
#include <hip/hip_runtime.h>
#include <rccl/rccl.h>
#include <cstdio>

#define HIP_CHECK(expr) do { hipError_t _e = (expr); if (_e != hipSuccess) { \
    fprintf(stderr, "HIP error %s at %s:%d\n", hipGetErrorString(_e), __FILE__, __LINE__); \
    return; } } while (0)

namespace agq {
#define LAS __attribute__((address_space(3)))
#define GAS __attribute__((address_space(1)))
typedef unsigned u32x4 __attribute__((ext_vector_type(4)));
typedef int i32x4 __attribute__((ext_vector_type(4)));
typedef float f32x32 __attribute__((ext_vector_type(32)));
typedef unsigned long long u64;

constexpr int NR = 8;
constexpr int MT = 4096, KK = 4096, NL = 2048, MS = MT / NR;
constexpr int NWG = 256, NGEMM = 208, NCOMM = NWG - NGEMM;
constexpr int NSUB = KK / 32;
constexpr int NKT = KK / 64;
constexpr int NPIECE = (MS / 64) * NSUB;
constexpr int S = 26;
constexpr int NP0 = 16;
constexpr int LEAD = 4;
constexpr int TAPER = 24, TAPER_M = (1 << 20) / (8 * TAPER + 4) + 1;
constexpr bool taper_ok() { for (int v = 0; v < NPIECE; ++v) if ((v * TAPER_M >> 20) != v / (8 * TAPER + 4)) return false; return LEAD * (TAPER + 1) <= NSUB && 8 * TAPER >= NP0 + S; }
static_assert(taper_ok(), "order of delivery");
constexpr unsigned NTICK = NPIECE - NP0 - S;
constexpr int HDR_EARLY = 16;
constexpr int POLL_LEAD = 10;
constexpr int SPEC_TH = 60, SPEC_N = 14;
static_assert(7 * S <= 4 * NCOMM && 4 * LEAD <= NP0 && NP0 <= S, "roles");
__device__ __forceinline__ int order_piece(int o) {
  const int v = o - 4 * LEAD;
  return v < 0 ? (o >> 2) * 8 + 4 + (o & 3) : v + (v & 4) * 2 * LEAD;
}

constexpr size_t AT_GROUP = (size_t)NSUB * 4096;
constexpr size_t WS_AT = 0;
constexpr size_t WS_PK = WS_AT + 64 * AT_GROUP;
constexpr unsigned PK_HI = NPIECE * 3072u, PK_BYTES = NPIECE * 5120u;
constexpr size_t WS_HDR = WS_PK + (6u << 20);
constexpr size_t WS_AFL = WS_HDR + (size_t)NR * NWG * 128;
constexpr size_t WS_AMX = WS_AFL + 16 * NSUB * 4 * 128;
constexpr size_t WS_AM2 = WS_AMX + NWG * 128;
constexpr size_t WS_SEQ = WS_AM2 + 2 * 48 * 128;
constexpr size_t WS_TCK = WS_SEQ + NWG * 128;
constexpr size_t WS_TMO = WS_TCK + NR * 128;
constexpr size_t WS_END = WS_TMO + 128;
__device__ __forceinline__ unsigned flag_off(int grp, int sub) { return (((grp >> 2) * NSUB + sub) * 4 + (grp & 3)) * 128; }

constexpr int NS = 6, SLOT_B = 20480, W_PASS = 2 * SLOT_B;
constexpr int CT_ST = NS * SLOT_B;
constexpr int CT_CALL = CT_ST + 64, CT_MAX = CT_CALL + 4, CT_MAXN = CT_CALL + 8, CT_MAXSENT = CT_CALL + 12;
constexpr int CT_AMAX = CT_CALL + 16;
constexpr int CT_PACKN = CT_CALL + 24, CT_HDRSENT = CT_CALL + 28, CT_HDR = CT_CALL + 32;
constexpr int CT_FREE = CT_ST + 256;
constexpr int CT_DUM = CT_FREE + 1024;
constexpr int LDS_B = CT_DUM + 10240;
constexpr unsigned DONE = 1000;
constexpr u64 DEADLINE = 80000000ull;

struct Params { const char* x; char* self; int rank; int pad; const char* w; char* out; char* ws[NR]; char* rot[NR]; };

__device__ __forceinline__ u64 rt() { return __builtin_amdgcn_s_memrealtime(); }
__device__ __forceinline__ unsigned rfl(unsigned v) { return __builtin_amdgcn_readfirstlane(v); }
__device__ __forceinline__ i32x4 mkr(const void* p, unsigned bytes) {
  u64 a = (u64)p; i32x4 r; r.x = rfl((unsigned)a); r.y = rfl((unsigned)(a >> 32)) & 0xffff; r.z = bytes; r.w = 0x00020000; return r;
}
__device__ __forceinline__ unsigned ld_sys(const char* p) { return __hip_atomic_load((GAS unsigned*)p, __ATOMIC_RELAXED, __HIP_MEMORY_SCOPE_SYSTEM); }
__device__ __forceinline__ u64 ld_sys64(const char* p) { return __hip_atomic_load((GAS u64*)p, __ATOMIC_RELAXED, __HIP_MEMORY_SCOPE_SYSTEM); }
__device__ __forceinline__ void st_granule(char* p, unsigned tag, unsigned v) {
  __hip_atomic_store((GAS u64*)p, ((u64)tag << 32) | v, __ATOMIC_RELAXED, __HIP_MEMORY_SCOPE_SYSTEM);
}
__device__ __forceinline__ unsigned ld_tmo(char* ws) { return __hip_atomic_load((GAS unsigned*)(ws + WS_TMO), __ATOMIC_RELAXED, __HIP_MEMORY_SCOPE_AGENT); }
__device__ __forceinline__ void st_tmo(char* ws, unsigned call) { __hip_atomic_store((GAS unsigned*)(ws + WS_TMO), call, __ATOMIC_RELAXED, __HIP_MEMORY_SCOPE_AGENT); }
__device__ __forceinline__ void lds_st(LAS char* p, unsigned v) {
  asm volatile("" ::: "memory"); __hip_atomic_store((LAS unsigned*)p, v, __ATOMIC_RELAXED, __HIP_MEMORY_SCOPE_WORKGROUP); asm volatile("" ::: "memory");
}
__device__ __forceinline__ unsigned lds_ld(LAS char* p) {
  const unsigned v = __hip_atomic_load((LAS unsigned*)p, __ATOMIC_RELAXED, __HIP_MEMORY_SCOPE_WORKGROUP); asm volatile("" ::: "memory"); return v;
}
__device__ __forceinline__ unsigned lds_inc(LAS char* p) {
  const unsigned v = __hip_atomic_fetch_add((LAS unsigned*)p, 1u, __ATOMIC_RELAXED, __HIP_MEMORY_SCOPE_WORKGROUP); asm volatile("" ::: "memory"); return v;
}
__device__ __forceinline__ unsigned umax(unsigned a, unsigned b) { return a > b ? a : b; }
__device__ __forceinline__ unsigned wave_max(unsigned v) {
  v = umax(v, __builtin_amdgcn_update_dpp(0u, v, 0xB1, 0xf, 0xf, true));
  v = umax(v, __builtin_amdgcn_update_dpp(0u, v, 0x4E, 0xf, 0xf, true));
  v = umax(v, __builtin_amdgcn_update_dpp(0u, v, 0x141, 0xf, 0xf, true));
  v = umax(v, __builtin_amdgcn_update_dpp(0u, v, 0x140, 0xf, 0xf, true));
  return umax(umax(__builtin_amdgcn_readlane(v, 0), __builtin_amdgcn_readlane(v, 16)),
              umax(__builtin_amdgcn_readlane(v, 32), __builtin_amdgcn_readlane(v, 48)));
}
#define ST16(base, voff, off, val) asm volatile("global_store_dwordx4 %0, %1, %2 offset:" #off " sc0 sc1\n\ts_nop 1" :: "v"(voff), "v"(val), "s"(base) : "memory")
__device__ __forceinline__ void st_flag(char* p, int lane, unsigned call) {
  const u32x4 v = {call, call, call, call};
  if (lane < 8) asm volatile("global_store_dwordx4 %0, %1, off sc0 sc1\n\ts_nop 1" :: "v"(p + lane * 16), "v"(v) : "memory");
}

__global__ void __launch_bounds__(512, 2) ag_gemm_q8(Params P) {
  extern __shared__ __attribute__((aligned(16))) char lds_[];
  LAS char* lds = (LAS char*)lds_;
  const int tid = threadIdx.x, lane = tid & 63, wv = rfl(tid >> 6);
  const int b = blockIdx.x;
  const int pid = b * 4 + (wv & 3);
  const int ps = 2 * (pid >> 4) + (pid & 1), pg = (pid >> 1) & 7;
  const i32x4 rx = mkr(P.x, MS * KK * 2);
#define XLOAD(C0, C1, C2, C3, A, B, C, D, sub, grp, SKIP) \
    asm volatile("s_cmp_lg_u32 %9, 0\n\ts_cbranch_scc1 1f\n\t" \
                 "buffer_load_dwordx4 %0, %4, %5, 0 offen\n\tbuffer_load_dwordx4 %1, %4, %5, %6 offen\n\t" \
                 "buffer_load_dwordx4 %2, %4, %5, %7 offen\n\tbuffer_load_dwordx4 %3, %4, %5, %8 offen\n1:" \
                 : C0(A), C1(B), C2(C), C3(D) \
                 : "v"((lane & 15) * (KK * 2) + (lane >> 4) * 16 + (grp) * (64 * KK * 2) + (sub) * 64), "s"(rx), \
                   "s"(16 * KK * 2), "s"(32 * KK * 2), "s"(48 * KK * 2), "s"(SKIP) : "memory", "scc")
  u32x4 R0, R1, R2, R3;
  const int late = (wv >> 2) | ((b + NCOMM) >> 8);
  XLOAD("={v[64:67]}", "={v[68:71]}", "={v[72:75]}", "={v[76:79]}", R0, R1, R2, R3, ps, pg, late);

  const int xcd = b & 7, bi = b >> 3;
  const bool comm = b >= NGEMM;
  const int ci = b - NGEMM, gw = wv * NCOMM + ci, idx = gw / 7;
  int rank = P.rank;
  char* ws = P.self;
  char* wsrc = P.rot[(gw % 7) & 7];
  asm volatile("" : "+s"(rank), "+s"(ws), "+s"(wsrc));
  const unsigned src = (unsigned)(rank + 1 + gw % 7) & 7u;
  const bool mover = comm && idx < S && src < (unsigned)NR;
  const bool packer = comm ? wv >= 4 : wv < 4;
  const int sl = (0x04372615 >> (4 * xcd)) & 7;
  const int rb = (2 * rank + ((sl & 3) == 0 ? (bi & 1) - (sl >> 1) : 2 * (2 * (sl & 3) - 1 + (bi & 1)) + (sl < 4))) & 15;
  const int cb = bi >> 1, c128 = cb * 128, c32 = 13 * 128 + cb * 32;
  const unsigned l16 = lane * 16;

  if (wv >= 4) lds_st(lds + CT_FREE + (tid - 256) * 4, 0);
  if (wv == 7) lds_st(lds + CT_ST + lane * 4, 0);
#define START_BARRIER() asm volatile("s_waitcnt lgkmcnt(0)\n\ts_barrier" ::: "memory")
  auto get_call = [&]() { unsigned c; while ((c = rfl(lds_ld(lds + CT_CALL))) == 0) __builtin_amdgcn_s_sleep(1); return c; };
  if (wv == 7) {
    unsigned v = 0;
    const unsigned zero = 0, one = 1;
    char* sp = ws + WS_SEQ + b * 128;
    asm volatile("s_nop 4\n\ts_mov_b64 exec, 1\n\tglobal_atomic_add %0, %1, %2, %3 sc0\n\ts_mov_b64 exec, -1\n\t"
                 "s_waitcnt lgkmcnt(0)\n\ts_barrier\n\ts_waitcnt vmcnt(0)"
                 : "+v"(v) : "v"(zero), "v"(one), "s"(sp) : "memory");
    if (lane == 0) lds_st(lds + CT_CALL, v + 1);
  }
  if (packer) {
    const int i = ps * 8 + pg, grp = rank * 8 + pg;
    XLOAD("+{v[64:67]}", "+{v[68:71]}", "+{v[72:75]}", "+{v[76:79]}", R0, R1, R2, R3, ps, pg, late ^ 1);
    char* peer = P.rot[(lane >> 2) & 7];
    if (wv != 7) START_BARRIER();
    char* pk = ws + WS_PK + (size_t)i * 3072;
    char* ph = ws + WS_PK + PK_HI + (size_t)i * 2048;
    char* at = ws + WS_AT + (size_t)grp * AT_GROUP + (size_t)ps * 4096;
    asm volatile("s_nop 4" : "+v"(peer) : "s"(pk), "s"(ph), "s"(at));
    asm volatile("s_waitcnt vmcnt(0)" : "+{v[64:67]}"(R0), "+{v[68:71]}"(R1), "+{v[72:75]}"(R2), "+{v[76:79]}"(R3) :: "memory");
    const u32x4 R[4] = {R0, R1, R2, R3};
    typedef unsigned short u16x2 __attribute__((ext_vector_type(2)));
    u16x2 pm = {0, 0};
#pragma unroll
    for (int m = 0; m < 4; ++m)
#pragma unroll
      for (int k = 0; k < 4; ++k) pm = __builtin_elementwise_max(pm, __builtin_bit_cast(u16x2, R[m][k] & 0x7fff7fffu));
    const unsigned emax = wave_max(pm.x > pm.y ? pm.x : pm.y) >> 7, of = ~emax & 1, top = (emax + of) >> 1;
    const unsigned of4 = of * 0x00800080u, top4 = top * 0x01010101u;
    u32x4 L[2], H[2];
#pragma unroll
    for (int j = 0; j < 8; ++j) {
      const unsigned r0 = R[j >> 1][2 * (j & 1)] + of4, r1 = R[j >> 1][2 * (j & 1) + 1] + of4;
      H[j >> 2][j & 3] = __builtin_amdgcn_perm(r1, r0, 0x07050301u); L[j >> 2][j & 3] = __builtin_amdgcn_perm(r1, r0, 0x06040200u);
    }
    ST16(pk, l16, 0, L[0]); ST16(pk, l16, 1024, L[1]);
    unsigned c0 = 0, c1 = 0, sg = 0, ex = 0;
    u32x4 Dv[2];
#pragma unroll
    for (int j = 0; j < 8; ++j) {
      const unsigned h = H[j >> 2][j & 3], d = top4 - (h & 0x7f7f7f7fu);
      if (j < 4) c0 |= (d & 0x03030303u) << (2 * j); else c1 |= (d & 0x03030303u) << (2 * j - 8);
      sg |= (h & 0x80808080u) >> j;
      ex |= ((d + 0x7c7c7c7cu) & 0x80808080u) >> (7 - j);
      Dv[j >> 2][j & 3] = d;
    }
    LAS char* sc = lds + 32768 + (wv & 3) * 4096;
    *(LAS u32x4*)(sc + 2048 + l16) = Dv[0]; *(LAS u32x4*)(sc + 3072 + l16) = Dv[1];
    *(LAS unsigned*)(sc + lane * 4) = c0; *(LAS unsigned*)(sc + 256 + lane * 4) = c1; *(LAS unsigned*)(sc + 512 + lane * 4) = sg;
    unsigned sum = __builtin_popcount(ex);
    const unsigned own = sum;
    sum += __builtin_amdgcn_update_dpp(0u, sum, 0x111, 0xf, 0xf, true);
    sum += __builtin_amdgcn_update_dpp(0u, sum, 0x112, 0xf, 0xf, true);
    sum += __builtin_amdgcn_update_dpp(0u, sum, 0x114, 0xf, 0xf, true);
    sum += __builtin_amdgcn_update_dpp(0u, sum, 0x118, 0xf, 0xf, true);
    sum += __builtin_amdgcn_update_dpp(0u, sum, 0x142, 0xa, 0xf, false);
    sum += __builtin_amdgcn_update_dpp(0u, sum, 0x143, 0xc, 0xf, false);
    const unsigned cnt = __builtin_amdgcn_readlane(sum, 63);
    unsigned slot = sum - own;
    asm volatile("" ::: "memory");
    if (cnt <= 128) {
      do {
        unsigned pos[3], val[3];
#pragma unroll
        for (int k = 0; k < 3; ++k) {
          const unsigned bit = __builtin_ctz(ex | 0x80000000u);
          pos[k] = ((bit & 4) << 8) + ((bit & 3) << 2) + (bit >> 3) + l16;
          val[k] = *(LAS unsigned char*)(sc + 2048 + pos[k]);
          const unsigned at2 = ex ? slot : 130 + 2 * lane;
          pos[k] |= at2 << 16;
          slot += ex != 0;
          ex &= ex - 1;
        }
#pragma unroll
        for (int k = 0; k < 3; ++k) *(LAS unsigned short*)(sc + 768 + (pos[k] >> 16) * 2) = (unsigned short)(pos[k] | ((val[k] >> 2) << 11));
      } while (__any(ex != 0));
    }
    const bool raw = cnt > 128;
    const unsigned hdr = top | (of << 7) | (raw ? 0x100u : cnt << 16);
    asm volatile("" ::: "memory");
    const u32x4 N = *(LAS u32x4*)(sc + l16);
    if (!raw) ST16(pk, l16, 2048, N);
    else { ST16(ph, l16, 0, H[0]); ST16(ph, l16, 1024, H[1]); }
    ST16(at, l16, 0, R[0]); ST16(at, l16, 1024, R[1]); ST16(at, l16, 2048, R[2]); ST16(at, l16, 3072, R[3]);
    const unsigned call = get_call();
    char* hp = peer + WS_HDR + ((size_t)rank * NWG + b) * 128 + (lane & 3) * 8;
    asm volatile("s_waitcnt vmcnt(4)" ::: "memory");
    if (b < HDR_EARLY && lane < 28 && (lane & 3) == (wv & 3)) st_granule(hp, call, hdr);
    unsigned tick = 0;
    if (lane == 0) { lds_st(lds + CT_HDR + (wv & 3) * 4, hdr); tick = lds_inc(lds + CT_PACKN); }
    const bool last = rfl(tick) == 3;
    asm volatile("s_waitcnt vmcnt(0)" ::: "memory");
    st_flag(ws + WS_AFL + flag_off(grp, ps), lane, call);
    if (last) {
      if (b >= HDR_EARLY && lane < 28) st_granule(hp, call, lds_ld(lds + CT_HDR + (lane & 3) * 4));
      lds_st(lds + CT_HDRSENT, 1);
    }
  } else if (mover && idx < NP0) {
    const int i = order_piece(idx), grp = rank * 8 + (i & 7);
    u32x4 Q0, Q1, Q2, Q3;
    XLOAD("={v[80:83]}", "={v[84:87]}", "={v[88:91]}", "={v[92:95]}", Q0, Q1, Q2, Q3, i >> 3, i & 7, 0);
    START_BARRIER();
    char* at = wsrc + WS_AT + (size_t)grp * AT_GROUP + (size_t)(i >> 3) * 4096;
    asm volatile("s_nop 4" :: "s"(at));
    asm volatile("s_waitcnt vmcnt(0)" : "+{v[80:83]}"(Q0), "+{v[84:87]}"(Q1), "+{v[88:91]}"(Q2), "+{v[92:95]}"(Q3) :: "memory");
    ST16(at, l16, 0, Q0); ST16(at, l16, 1024, Q1); ST16(at, l16, 2048, Q2); ST16(at, l16, 3072, Q3);
    const unsigned call = get_call();
    asm volatile("s_waitcnt vmcnt(0)" ::: "memory");
    st_flag(wsrc + WS_AFL + flag_off(grp, i >> 3), lane, call);
  } else if (wv != 7) START_BARRIER();
  const unsigned call = get_call();

  if (comm) {
    if (wv == 7 && ci < 6) {
      const bool fin = ci < 4;
      char* dst = P.ws[lane & 7] + WS_AM2 + ((call & 1) * 48 + (fin ? rank * 4 + ci : 32 + rank * 2 + ci - 4)) * 128;
      const char* ap = ws + WS_AMX + ((fin ? ci : 2 * (ci - 4)) * 52 + (lane < 52 ? lane : 0)) * 128;
      const u64 t0 = rt();
      unsigned sent = 0;
      u64 v = 0, v2 = 0;
      for (int spins = 1;; ++spins) {
        if (lane < 52 && (unsigned)(v >> 32) < call) v = ld_sys64(ap);
        if (!fin && lane < 52 && (unsigned)(v2 >> 32) < call) v2 = ld_sys64(ap + 52 * 128);
        const bool ok = (unsigned)(v >> 32) >= call, ok2 = (unsigned)(v2 >> 32) >= call;
        const int n = __builtin_popcountll(__ballot(ok)) + __builtin_popcountll(__ballot(ok2));
        if (!fin && n == 104) break;
        if (n >= (fin ? 52 : SPEC_TH)) {
          const unsigned gm = wave_max(umax(ok ? (unsigned)v : 0u, ok2 ? (unsigned)v2 : 0u));
          if (gm >= sent) { if (lane < NR) st_granule(dst, call, gm); sent = gm + 1; }
          if (fin) break;
        }
        if ((spins & 31) == 0 && (rfl(ld_tmo(ws)) == call || rt() - t0 > DEADLINE)) { st_tmo(ws, call); break; }
        if (n < 8) __builtin_amdgcn_s_sleep(16);
      }
    }
    if (mover) {
      const unsigned i0 = order_piece(NP0 + idx);
      const char* phd = ws + WS_HDR + (size_t)src * NWG * 128;
      const char* ppk = wsrc + WS_PK;
      char* pat = ws + WS_AT + (size_t)src * 8 * AT_GROUP;
      char* pfl = ws + WS_AFL + flag_off(src * 8, 0);
      char* ptc = ws + WS_TCK + src * 128;
      const unsigned bias = NP0 - 4 * LEAD + 2 * S - (call - 1) * NTICK, tk0 = NP0 - 4 * LEAD + S + idx - bias;
      const unsigned t0 = (unsigned)rt(), dl32 = (unsigned)DEADLINE, pld = wv * 3072;
      unsigned pst;
      asm volatile(
        "s_mov_b64 s[40:41], %[pk]\ns_and_b32 s41, s41, 0xffff\ns_mov_b32 s43, 0x20000\ns_mov_b64 s[44:45], %[at]\ns_mov_b64 s[46:47], %[fl]\n"
        "s_mov_b64 s[48:49], %[hd]\ns_mov_b64 s[50:51], %[tc]\ns_mov_b32 s54, -1\ns_mov_b32 s55, %[p1]\ns_mov_b32 s78, 0x03030303\ns_mov_b32 s79, 0x80808080\n"
        "s_mov_b32 s80, 0x05010400\ns_mov_b32 s81, 0x07030602\nv_mov_b32 v179, %[l16]\nv_bfrev_b32 v180, 1\nv_lshrrev_b32 v188, 4, %[l16]\n"
        "v_add_u32 v189, %[ld], v179\nv_lshl_add_u32 v190, v188, 2, %[ld]\nv_lshl_add_u32 v191, v188, 1, %[ld]\nv_mov_b32 v185, 0\nv_mov_b32 v186, 1\n"
        "v_mov_b32 v194, %[call]\nv_mov_b32 v195, %[call]\nv_mov_b32 v196, %[call]\nv_mov_b32 v197, %[call]\nv_mov_b32 v184, %[tk0]\nv_mov_b32 v182, 0\n"
        "v_mov_b32 v183, 0\nv_mov_b32 v164, 0\nv_mov_b32 v165, 0\nv_mov_b32 v166, 0\nv_mov_b32 v167, 0\nds_write_b128 v189, v[164:167] offset:1024\n"
        "ds_write_b128 v189, v[164:167] offset:2048\ns_add_u32 s61, %[ld], 1024\ns_mov_b32 %[st], 0\nLp0_%=:\nv_readfirstlane_b32 s64, v182\n"
        "v_readfirstlane_b32 s65, v183\nv_readfirstlane_b32 s56, v184\ns_cmp_ge_u32 s55, 1024\ns_cbranch_scc1 Ln0_%=\ns_cmp_lt_u32 s65, %[call]\n"
        "s_cbranch_scc1 Lw0_%=\ns_mov_b32 s42, 0x500000\ns_mul_i32 s67, s55, 3072\nbuffer_load_dwordx4 v[116:119], v179, s[40:43], s67 offen sc0 sc1\n"
        "buffer_load_dwordx4 v[120:123], v179, s[40:43], s67 offen offset:1024 sc0 sc1\ns_bitcmp1_b32 s64, 8\ns_cbranch_scc1 Lq0_%=\n"
        "s_bfe_u32 s66, s64, 0x80010\ns_add_u32 s66, s66, 391\ns_lshr_b32 s66, s66, 3\nv_cmp_gt_u32 vcc, s66, v188\ns_nop 1\n"
        "v_cndmask_b32 v181, v180, v179, vcc\nbuffer_load_dwordx4 v[124:127], v181, s[40:43], s67 offen offset:2048 sc0 sc1\nLn0_%=:\n"
        "s_add_u32 s56, s56, %[bias]\ns_cmp_ge_u32 s56, %[nv]\ns_cselect_b32 s56, -2, s56\ns_cbranch_scc1 Lt0_%=\ns_mul_i32 s66, s56, %[mg]\n"
        "s_lshr_b32 s66, s66, 20\ns_min_u32 s66, s66, %[ll]\ns_lshl_b32 s66, s66, 2\ns_sub_u32 s69, %[l8], s66\ns_add_u32 s68, s56, s66\ns_bitcmp1_b32 s68, 2\n"
        "s_cselect_b32 s66, s69, s66\ns_add_u32 s56, s56, s66\ns_bfe_u32 s69, s56, 0x20001\ns_lshr_b32 s66, s56, 4\ns_lshl2_add_u32 s69, s66, s69\n"
        "s_lshl_b32 s69, s69, 7\ns_and_b32 s66, s56, 8\ns_or_b32 s69, s69, s66\ns_and_b32 s66, s56, 1\ns_lshl4_add_u32 s66, s66, s69\ns_add_u32 s70, s48, s66\n"
        "s_addc_u32 s71, s49, 0\ns_mov_b64 exec, 1\nglobal_load_dwordx2 v[182:183], v185, s[70:71] sc0 sc1\nglobal_atomic_add v184, v185, v186, s[50:51] sc0\n"
        "s_mov_b64 exec, -1\nLt0_%=:\ns_cmp_eq_u32 s54, -1\ns_cbranch_scc1 Ls0_%=\ns_bitcmp1_b32 s57, 8\ns_cbranch_scc1 Lu0_%=\n"
        "ds_write_b128 v189, v[108:111]\nds_read_u16 v175, v191 offset:768\nds_read_u16 v176, v191 offset:896\nds_read2_b32 v[172:173], v190 offset1:64\n"
        "ds_read_b32 v174, v190 offset:512\ns_bfe_u32 s66, s57, 0x80010\nv_cmp_gt_u32 s[58:59], s66, v188\ns_sub_u32 s66, s66, 64\n"
        "v_cmp_gt_i32 s[62:63], s66, v188\ns_and_b32 s60, s57, 0x7f\ns_mul_i32 s60, s60, 0x01010101\ns_waitcnt lgkmcnt(0)\nv_and_b32 v177, 0x7ff, v175\n"
        "v_and_b32 v178, 0x7ff, v176\nv_lshrrev_b32 v175, 11, v175\nv_lshrrev_b32 v176, 11, v176\nv_add_u32 v177, s61, v177\nv_add_u32 v178, s61, v178\n"
        "v_lshlrev_b32 v175, 2, v175\nv_lshlrev_b32 v176, 2, v176\ns_mov_b64 exec, s[58:59]\nds_write_b8 v177, v175\ns_mov_b64 exec, s[62:63]\n"
        "ds_write_b8 v178, v176\ns_mov_b64 exec, -1\nds_read_b128 v[148:151], v189 offset:1024\nds_read_b128 v[152:155], v189 offset:2048\n"
        "v_lshrrev_b32 v157, 2, v172\nv_lshlrev_b32 v165, 1, v174\nv_lshrrev_b32 v158, 4, v172\nv_lshlrev_b32 v166, 2, v174\nv_lshrrev_b32 v159, 6, v172\n"
        "v_lshlrev_b32 v167, 3, v174\nv_lshlrev_b32 v168, 4, v174\nv_lshrrev_b32 v161, 2, v173\nv_lshlrev_b32 v169, 5, v174\nv_lshrrev_b32 v162, 4, v173\n"
        "v_lshlrev_b32 v170, 6, v174\nv_lshrrev_b32 v163, 6, v173\nv_lshlrev_b32 v171, 7, v174\ns_waitcnt lgkmcnt(0)\ns_mov_b64 exec, s[58:59]\n"
        "ds_write_b8 v177, v185\ns_mov_b64 exec, s[62:63]\nds_write_b8 v178, v185\ns_mov_b64 exec, -1\nv_and_or_b32 v156, v172, s78, v148\n"
        "v_and_or_b32 v157, v157, s78, v149\nv_and_or_b32 v158, v158, s78, v150\nv_and_or_b32 v159, v159, s78, v151\nv_and_or_b32 v160, v173, s78, v152\n"
        "v_and_or_b32 v161, v161, s78, v153\nv_and_or_b32 v162, v162, s78, v154\nv_and_or_b32 v163, v163, s78, v155\nv_sub_u32 v156, s60, v156\n"
        "v_sub_u32 v157, s60, v157\nv_sub_u32 v158, s60, v158\nv_sub_u32 v159, s60, v159\nv_sub_u32 v160, s60, v160\nv_sub_u32 v161, s60, v161\n"
        "v_sub_u32 v162, s60, v162\nv_sub_u32 v163, s60, v163\nv_and_or_b32 v156, v174, s79, v156\nv_and_or_b32 v157, v165, s79, v157\n"
        "v_and_or_b32 v158, v166, s79, v158\nv_and_or_b32 v159, v167, s79, v159\nv_and_or_b32 v160, v168, s79, v160\nv_and_or_b32 v161, v169, s79, v161\n"
        "v_and_or_b32 v162, v170, s79, v162\nv_and_or_b32 v163, v171, s79, v163\nLv0_%=:\ns_and_b32 s66, s54, 7\ns_lshr_b32 s67, s54, 3\n"
        "s_lshl_b32 s68, s66, 19\ns_lshl_b32 s69, s67, 12\ns_add_u32 s68, s68, s69\ns_add_u32 s70, s44, s68\ns_addc_u32 s71, s45, 0\ns_and_b32 s60, s57, 0x80\n"
        "s_mul_i32 s60, s60, 0x10001\nv_perm_b32 v132, v156, v100, s80\nv_perm_b32 v133, v156, v100, s81\nv_perm_b32 v134, v157, v101, s80\n"
        "v_perm_b32 v135, v157, v101, s81\nv_perm_b32 v136, v158, v102, s80\nv_perm_b32 v137, v158, v102, s81\nv_perm_b32 v138, v159, v103, s80\n"
        "v_perm_b32 v139, v159, v103, s81\nv_subrev_u32 v132, s60, v132\nv_subrev_u32 v133, s60, v133\nv_subrev_u32 v134, s60, v134\n"
        "v_subrev_u32 v135, s60, v135\nv_subrev_u32 v136, s60, v136\nv_subrev_u32 v137, s60, v137\nv_subrev_u32 v138, s60, v138\nv_subrev_u32 v139, s60, v139\n"
        "global_store_dwordx4 v179, v[132:135], s[70:71] offset:0 sc0 sc1\nglobal_store_dwordx4 v179, v[136:139], s[70:71] offset:1024 sc0 sc1\n"
        "v_perm_b32 v140, v160, v104, s80\nv_perm_b32 v141, v160, v104, s81\nv_perm_b32 v142, v161, v105, s80\nv_perm_b32 v143, v161, v105, s81\n"
        "v_perm_b32 v144, v162, v106, s80\nv_perm_b32 v145, v162, v106, s81\nv_perm_b32 v146, v163, v107, s80\nv_perm_b32 v147, v163, v107, s81\n"
        "v_subrev_u32 v140, s60, v140\nv_subrev_u32 v141, s60, v141\nv_subrev_u32 v142, s60, v142\nv_subrev_u32 v143, s60, v143\nv_subrev_u32 v144, s60, v144\n"
        "v_subrev_u32 v145, s60, v145\nv_subrev_u32 v146, s60, v146\nv_subrev_u32 v147, s60, v147\n"
        "global_store_dwordx4 v179, v[140:143], s[70:71] offset:2048 sc0 sc1\nglobal_store_dwordx4 v179, v[144:147], s[70:71] offset:3072 sc0 sc1\n"
        "s_lshr_b32 s68, s66, 2\ns_lshl_b32 s68, s68, 9\ns_and_b32 s66, s66, 3\ns_lshl2_add_u32 s66, s67, s66\ns_add_u32 s66, s66, s68\n"
        "s_lshl_b32 s66, s66, 7\ns_add_u32 s72, s46, s66\ns_addc_u32 s73, s47, 0\ns_waitcnt vmcnt(0)\ns_mov_b64 exec, 0xff\n"
        "global_store_dwordx4 v179, v[194:197], s[72:73] sc0 sc1\ns_mov_b64 exec, -1\ns_branch Lr0_%=\nLs0_%=:\ns_waitcnt vmcnt(0)\nLr0_%=:\n"
        "s_mov_b32 s54, s55\ns_mov_b32 s55, s56\ns_mov_b32 s57, s64\ns_cmp_ge_u32 s54, 1024\ns_cbranch_scc1 Le_%=\nLp1_%=:\nv_readfirstlane_b32 s64, v182\n"
        "v_readfirstlane_b32 s65, v183\nv_readfirstlane_b32 s56, v184\ns_cmp_ge_u32 s55, 1024\ns_cbranch_scc1 Ln1_%=\ns_cmp_lt_u32 s65, %[call]\n"
        "s_cbranch_scc1 Lw1_%=\ns_mov_b32 s42, 0x500000\ns_mul_i32 s67, s55, 3072\nbuffer_load_dwordx4 v[100:103], v179, s[40:43], s67 offen sc0 sc1\n"
        "buffer_load_dwordx4 v[104:107], v179, s[40:43], s67 offen offset:1024 sc0 sc1\ns_bitcmp1_b32 s64, 8\ns_cbranch_scc1 Lq1_%=\n"
        "s_bfe_u32 s66, s64, 0x80010\ns_add_u32 s66, s66, 391\ns_lshr_b32 s66, s66, 3\nv_cmp_gt_u32 vcc, s66, v188\ns_nop 1\n"
        "v_cndmask_b32 v181, v180, v179, vcc\nbuffer_load_dwordx4 v[108:111], v181, s[40:43], s67 offen offset:2048 sc0 sc1\nLn1_%=:\n"
        "s_add_u32 s56, s56, %[bias]\ns_cmp_ge_u32 s56, %[nv]\ns_cselect_b32 s56, -2, s56\ns_cbranch_scc1 Lt1_%=\ns_mul_i32 s66, s56, %[mg]\n"
        "s_lshr_b32 s66, s66, 20\ns_min_u32 s66, s66, %[ll]\ns_lshl_b32 s66, s66, 2\ns_sub_u32 s69, %[l8], s66\ns_add_u32 s68, s56, s66\ns_bitcmp1_b32 s68, 2\n"
        "s_cselect_b32 s66, s69, s66\ns_add_u32 s56, s56, s66\ns_bfe_u32 s69, s56, 0x20001\ns_lshr_b32 s66, s56, 4\ns_lshl2_add_u32 s69, s66, s69\n"
        "s_lshl_b32 s69, s69, 7\ns_and_b32 s66, s56, 8\ns_or_b32 s69, s69, s66\ns_and_b32 s66, s56, 1\ns_lshl4_add_u32 s66, s66, s69\ns_add_u32 s70, s48, s66\n"
        "s_addc_u32 s71, s49, 0\ns_mov_b64 exec, 1\nglobal_load_dwordx2 v[182:183], v185, s[70:71] sc0 sc1\nglobal_atomic_add v184, v185, v186, s[50:51] sc0\n"
        "s_mov_b64 exec, -1\nLt1_%=:\ns_bitcmp1_b32 s57, 8\ns_cbranch_scc1 Lu1_%=\nds_write_b128 v189, v[124:127]\nds_read_u16 v175, v191 offset:768\n"
        "ds_read_u16 v176, v191 offset:896\nds_read2_b32 v[172:173], v190 offset1:64\nds_read_b32 v174, v190 offset:512\ns_bfe_u32 s66, s57, 0x80010\n"
        "v_cmp_gt_u32 s[58:59], s66, v188\ns_sub_u32 s66, s66, 64\nv_cmp_gt_i32 s[62:63], s66, v188\ns_and_b32 s60, s57, 0x7f\ns_mul_i32 s60, s60, 0x01010101\n"
        "s_waitcnt lgkmcnt(0)\nv_and_b32 v177, 0x7ff, v175\nv_and_b32 v178, 0x7ff, v176\nv_lshrrev_b32 v175, 11, v175\nv_lshrrev_b32 v176, 11, v176\n"
        "v_add_u32 v177, s61, v177\nv_add_u32 v178, s61, v178\nv_lshlrev_b32 v175, 2, v175\nv_lshlrev_b32 v176, 2, v176\ns_mov_b64 exec, s[58:59]\n"
        "ds_write_b8 v177, v175\ns_mov_b64 exec, s[62:63]\nds_write_b8 v178, v176\ns_mov_b64 exec, -1\nds_read_b128 v[148:151], v189 offset:1024\n"
        "ds_read_b128 v[152:155], v189 offset:2048\nv_lshrrev_b32 v157, 2, v172\nv_lshlrev_b32 v165, 1, v174\nv_lshrrev_b32 v158, 4, v172\n"
        "v_lshlrev_b32 v166, 2, v174\nv_lshrrev_b32 v159, 6, v172\nv_lshlrev_b32 v167, 3, v174\nv_lshlrev_b32 v168, 4, v174\nv_lshrrev_b32 v161, 2, v173\n"
        "v_lshlrev_b32 v169, 5, v174\nv_lshrrev_b32 v162, 4, v173\nv_lshlrev_b32 v170, 6, v174\nv_lshrrev_b32 v163, 6, v173\nv_lshlrev_b32 v171, 7, v174\n"
        "s_waitcnt lgkmcnt(0)\ns_mov_b64 exec, s[58:59]\nds_write_b8 v177, v185\ns_mov_b64 exec, s[62:63]\nds_write_b8 v178, v185\ns_mov_b64 exec, -1\n"
        "v_and_or_b32 v156, v172, s78, v148\nv_and_or_b32 v157, v157, s78, v149\nv_and_or_b32 v158, v158, s78, v150\nv_and_or_b32 v159, v159, s78, v151\n"
        "v_and_or_b32 v160, v173, s78, v152\nv_and_or_b32 v161, v161, s78, v153\nv_and_or_b32 v162, v162, s78, v154\nv_and_or_b32 v163, v163, s78, v155\n"
        "v_sub_u32 v156, s60, v156\nv_sub_u32 v157, s60, v157\nv_sub_u32 v158, s60, v158\nv_sub_u32 v159, s60, v159\nv_sub_u32 v160, s60, v160\n"
        "v_sub_u32 v161, s60, v161\nv_sub_u32 v162, s60, v162\nv_sub_u32 v163, s60, v163\nv_and_or_b32 v156, v174, s79, v156\n"
        "v_and_or_b32 v157, v165, s79, v157\nv_and_or_b32 v158, v166, s79, v158\nv_and_or_b32 v159, v167, s79, v159\nv_and_or_b32 v160, v168, s79, v160\n"
        "v_and_or_b32 v161, v169, s79, v161\nv_and_or_b32 v162, v170, s79, v162\nv_and_or_b32 v163, v171, s79, v163\nLv1_%=:\ns_and_b32 s66, s54, 7\n"
        "s_lshr_b32 s67, s54, 3\ns_lshl_b32 s68, s66, 19\ns_lshl_b32 s69, s67, 12\ns_add_u32 s68, s68, s69\ns_add_u32 s70, s44, s68\ns_addc_u32 s71, s45, 0\n"
        "s_and_b32 s60, s57, 0x80\ns_mul_i32 s60, s60, 0x10001\nv_perm_b32 v132, v156, v116, s80\nv_perm_b32 v133, v156, v116, s81\n"
        "v_perm_b32 v134, v157, v117, s80\nv_perm_b32 v135, v157, v117, s81\nv_perm_b32 v136, v158, v118, s80\nv_perm_b32 v137, v158, v118, s81\n"
        "v_perm_b32 v138, v159, v119, s80\nv_perm_b32 v139, v159, v119, s81\nv_subrev_u32 v132, s60, v132\nv_subrev_u32 v133, s60, v133\n"
        "v_subrev_u32 v134, s60, v134\nv_subrev_u32 v135, s60, v135\nv_subrev_u32 v136, s60, v136\nv_subrev_u32 v137, s60, v137\nv_subrev_u32 v138, s60, v138\n"
        "v_subrev_u32 v139, s60, v139\nglobal_store_dwordx4 v179, v[132:135], s[70:71] offset:0 sc0 sc1\n"
        "global_store_dwordx4 v179, v[136:139], s[70:71] offset:1024 sc0 sc1\nv_perm_b32 v140, v160, v120, s80\nv_perm_b32 v141, v160, v120, s81\n"
        "v_perm_b32 v142, v161, v121, s80\nv_perm_b32 v143, v161, v121, s81\nv_perm_b32 v144, v162, v122, s80\nv_perm_b32 v145, v162, v122, s81\n"
        "v_perm_b32 v146, v163, v123, s80\nv_perm_b32 v147, v163, v123, s81\nv_subrev_u32 v140, s60, v140\nv_subrev_u32 v141, s60, v141\n"
        "v_subrev_u32 v142, s60, v142\nv_subrev_u32 v143, s60, v143\nv_subrev_u32 v144, s60, v144\nv_subrev_u32 v145, s60, v145\nv_subrev_u32 v146, s60, v146\n"
        "v_subrev_u32 v147, s60, v147\nglobal_store_dwordx4 v179, v[140:143], s[70:71] offset:2048 sc0 sc1\n"
        "global_store_dwordx4 v179, v[144:147], s[70:71] offset:3072 sc0 sc1\ns_lshr_b32 s68, s66, 2\ns_lshl_b32 s68, s68, 9\ns_and_b32 s66, s66, 3\n"
        "s_lshl2_add_u32 s66, s67, s66\ns_add_u32 s66, s66, s68\ns_lshl_b32 s66, s66, 7\ns_add_u32 s72, s46, s66\ns_addc_u32 s73, s47, 0\ns_waitcnt vmcnt(0)\n"
        "s_mov_b64 exec, 0xff\nglobal_store_dwordx4 v179, v[194:197], s[72:73] sc0 sc1\ns_mov_b64 exec, -1\ns_mov_b32 s54, s55\ns_mov_b32 s55, s56\n"
        "s_mov_b32 s57, s64\ns_cmp_ge_u32 s54, 1024\ns_cbranch_scc0 Lp0_%=\ns_branch Le_%=\nLw0_%=:\ns_bfe_u32 s69, s55, 0x20001\ns_lshr_b32 s66, s55, 4\n"
        "s_lshl2_add_u32 s69, s66, s69\ns_lshl_b32 s69, s69, 7\ns_and_b32 s66, s55, 8\ns_or_b32 s69, s69, s66\ns_and_b32 s66, s55, 1\n"
        "s_lshl4_add_u32 s66, s66, s69\ns_add_u32 s70, s48, s66\ns_addc_u32 s71, s49, 0\ns_waitcnt vmcnt(0)\n"
        "global_load_dwordx2 v[198:199], v185, s[70:71] sc0 sc1\ns_sleep 2\nglobal_load_dwordx2 v[200:201], v185, s[70:71] sc0 sc1\ns_sleep 2\n"
        "global_load_dwordx2 v[202:203], v185, s[70:71] sc0 sc1\ns_sleep 2\nLx0_%=:\nglobal_load_dwordx2 v[204:205], v185, s[70:71] sc0 sc1\n"
        "s_waitcnt vmcnt(3)\nv_readfirstlane_b32 s65, v199\nv_mov_b32 v182, v198\nv_mov_b32 v183, v199\ns_cmp_ge_u32 s65, %[call]\ns_cbranch_scc1 Lp0_%=\n"
        "s_sleep 2\nglobal_load_dwordx2 v[198:199], v185, s[70:71] sc0 sc1\ns_waitcnt vmcnt(3)\nv_readfirstlane_b32 s65, v201\nv_mov_b32 v182, v200\n"
        "v_mov_b32 v183, v201\ns_cmp_ge_u32 s65, %[call]\ns_cbranch_scc1 Lp0_%=\ns_sleep 2\nglobal_load_dwordx2 v[200:201], v185, s[70:71] sc0 sc1\n"
        "s_waitcnt vmcnt(3)\nv_readfirstlane_b32 s65, v203\nv_mov_b32 v182, v202\nv_mov_b32 v183, v203\ns_cmp_ge_u32 s65, %[call]\ns_cbranch_scc1 Lp0_%=\n"
        "s_sleep 2\nglobal_load_dwordx2 v[202:203], v185, s[70:71] sc0 sc1\ns_waitcnt vmcnt(3)\nv_readfirstlane_b32 s65, v205\nv_mov_b32 v182, v204\n"
        "v_mov_b32 v183, v205\ns_cmp_ge_u32 s65, %[call]\ns_cbranch_scc1 Lp0_%=\ns_sleep 2\ns_memrealtime s[76:77]\ns_waitcnt lgkmcnt(0)\n"
        "s_sub_u32 s76, s76, %[t0]\ns_cmp_lt_u32 s76, %[dl]\ns_cbranch_scc1 Lx0_%=\ns_mov_b32 %[st], 1\ns_branch Le_%=\nLq0_%=:\ns_lshl_b32 s68, s55, 11\n"
        "s_add_u32 s68, s68, 0x300000\nbuffer_load_dwordx4 v[124:127], v179, s[40:43], s68 offen sc0 sc1\n"
        "buffer_load_dwordx4 v[128:131], v179, s[40:43], s68 offen offset:1024 sc0 sc1\ns_branch Ln0_%=\nLu0_%=:\nv_mov_b32 v156, v108\nv_mov_b32 v157, v109\n"
        "v_mov_b32 v158, v110\nv_mov_b32 v159, v111\nv_mov_b32 v160, v112\nv_mov_b32 v161, v113\nv_mov_b32 v162, v114\nv_mov_b32 v163, v115\ns_branch Lv0_%=\n"
        "Lw1_%=:\ns_bfe_u32 s69, s55, 0x20001\ns_lshr_b32 s66, s55, 4\ns_lshl2_add_u32 s69, s66, s69\ns_lshl_b32 s69, s69, 7\ns_and_b32 s66, s55, 8\n"
        "s_or_b32 s69, s69, s66\ns_and_b32 s66, s55, 1\ns_lshl4_add_u32 s66, s66, s69\ns_add_u32 s70, s48, s66\ns_addc_u32 s71, s49, 0\ns_waitcnt vmcnt(0)\n"
        "global_load_dwordx2 v[198:199], v185, s[70:71] sc0 sc1\ns_sleep 2\nglobal_load_dwordx2 v[200:201], v185, s[70:71] sc0 sc1\ns_sleep 2\n"
        "global_load_dwordx2 v[202:203], v185, s[70:71] sc0 sc1\ns_sleep 2\nLx1_%=:\nglobal_load_dwordx2 v[204:205], v185, s[70:71] sc0 sc1\n"
        "s_waitcnt vmcnt(3)\nv_readfirstlane_b32 s65, v199\nv_mov_b32 v182, v198\nv_mov_b32 v183, v199\ns_cmp_ge_u32 s65, %[call]\ns_cbranch_scc1 Lp1_%=\n"
        "s_sleep 2\nglobal_load_dwordx2 v[198:199], v185, s[70:71] sc0 sc1\ns_waitcnt vmcnt(3)\nv_readfirstlane_b32 s65, v201\nv_mov_b32 v182, v200\n"
        "v_mov_b32 v183, v201\ns_cmp_ge_u32 s65, %[call]\ns_cbranch_scc1 Lp1_%=\ns_sleep 2\nglobal_load_dwordx2 v[200:201], v185, s[70:71] sc0 sc1\n"
        "s_waitcnt vmcnt(3)\nv_readfirstlane_b32 s65, v203\nv_mov_b32 v182, v202\nv_mov_b32 v183, v203\ns_cmp_ge_u32 s65, %[call]\ns_cbranch_scc1 Lp1_%=\n"
        "s_sleep 2\nglobal_load_dwordx2 v[202:203], v185, s[70:71] sc0 sc1\ns_waitcnt vmcnt(3)\nv_readfirstlane_b32 s65, v205\nv_mov_b32 v182, v204\n"
        "v_mov_b32 v183, v205\ns_cmp_ge_u32 s65, %[call]\ns_cbranch_scc1 Lp1_%=\ns_sleep 2\ns_memrealtime s[76:77]\ns_waitcnt lgkmcnt(0)\n"
        "s_sub_u32 s76, s76, %[t0]\ns_cmp_lt_u32 s76, %[dl]\ns_cbranch_scc1 Lx1_%=\ns_mov_b32 %[st], 1\ns_branch Le_%=\nLq1_%=:\ns_lshl_b32 s68, s55, 11\n"
        "s_add_u32 s68, s68, 0x300000\nbuffer_load_dwordx4 v[108:111], v179, s[40:43], s68 offen sc0 sc1\n"
        "buffer_load_dwordx4 v[112:115], v179, s[40:43], s68 offen offset:1024 sc0 sc1\ns_branch Ln1_%=\nLu1_%=:\nv_mov_b32 v156, v124\nv_mov_b32 v157, v125\n"
        "v_mov_b32 v158, v126\nv_mov_b32 v159, v127\nv_mov_b32 v160, v128\nv_mov_b32 v161, v129\nv_mov_b32 v162, v130\nv_mov_b32 v163, v131\ns_branch Lv1_%=\n"
        "Le_%=:\ns_waitcnt vmcnt(0) lgkmcnt(0)\n"
        : [st] "=&s"(pst)
        : [pk] "s"(ppk), [at] "s"(pat), [fl] "s"(pfl), [hd] "s"(phd), [tc] "s"(ptc), [p1] "s"(i0), [call] "s"(call), [bias] "s"(bias),
          [tk0] "s"(tk0), [t0] "s"(t0), [dl] "s"(dl32), [ld] "s"(pld), [nv] "i"(NPIECE - 4 * LEAD), [mg] "i"(TAPER_M), [ll] "i"(LEAD), [l8] "i"(8 * LEAD), [l16] "v"(l16)
        : "memory", "scc", "vcc",
          "s40", "s41", "s42", "s43", "s44", "s45", "s46", "s47", "s48", "s49", "s50", "s51", "s52", "s53", "s54", "s55",
          "s56", "s57", "s58", "s59", "s60", "s61", "s62", "s63", "s64", "s65", "s66", "s67", "s68", "s69", "s70", "s71",
          "s72", "s73", "s74", "s75", "s76", "s77", "s78", "s79", "s80", "s81", "s82", "s83", "s84", "s85", "v100", "v101",
          "v102", "v103", "v104", "v105", "v106", "v107", "v108", "v109", "v110", "v111", "v112", "v113", "v114", "v115", "v116", "v117",
          "v118", "v119", "v120", "v121", "v122", "v123", "v124", "v125", "v126", "v127", "v128", "v129", "v130", "v131", "v132", "v133",
          "v134", "v135", "v136", "v137", "v138", "v139", "v140", "v141", "v142", "v143", "v144", "v145", "v146", "v147", "v148", "v149",
          "v150", "v151", "v152", "v153", "v154", "v155", "v156", "v157", "v158", "v159", "v160", "v161", "v162", "v163", "v164", "v165",
          "v166", "v167", "v168", "v169", "v170", "v171", "v172", "v173", "v174", "v175", "v176", "v177", "v178", "v179", "v180", "v181",
          "v182", "v183", "v184", "v185", "v186", "v187", "v188", "v189", "v190", "v191", "v192", "v193", "v194", "v195", "v196", "v197",
          "v198", "v199", "v200", "v201", "v202", "v203", "v204", "v205");
      if (pst) st_tmo(ws, call);
    }
    __builtin_amdgcn_endpgm();
  }

  if (wv >= 4) while (rfl(lds_ld(lds + CT_HDRSENT)) == 0) __builtin_amdgcn_s_sleep(2);
  if (wv == 4) {
    const i32x4 rw = mkr(P.w, (unsigned)KK * NL * 2);
    const int q = lane >> 4, g1 = (lane >> 3) & 1, u = (lane >> 1) & 3;
    const unsigned voff = (8 * g1 + q) * (NL * 2) + 32 * (u ^ q) + 16 * (lane & 1);
    const unsigned voff2 = cb == 12 ? 0x80000000u
        : (8 * ((lane >> 4) & 1) + 4 * (lane >> 5) + ((lane >> 2) & 3)) * (NL * 2) + 32 * (((lane >> 1) ^ (lane >> 4)) & 1) + 16 * (lane & 1);
    LAS char* fullp = lds + CT_ST + 8 * (lane & 3);
    LAS char* freep = lds + CT_FREE + 256 * (lane & 3);
    int pub = 0;
    for (int kt = 0; kt < NKT; ++kt) {
      if (kt >= NS) {
        const unsigned need = kt - NS + 1;
        for (;;) {
          if (__all(lds_ld(freep) >= need)) break;
          if (pub < kt) { asm volatile("s_waitcnt vmcnt(0)" ::: "memory"); pub = kt; lds_st(fullp, pub); }
          __builtin_amdgcn_s_sleep(4);
        }
      }
      const unsigned ldsa = (kt % NS) * SLOT_B;
      unsigned soff = c128 * 2 + (unsigned)kt * 64 * (NL * 2), soff2 = (cb == 12 ? 0 : c32 * 2) + (unsigned)kt * 64 * (NL * 2), keep;
#define DMA2(RADD) "buffer_load_dwordx4 %[vo], %[rw], %[so] offen lds\n s_add_u32 m0, m0, 0x400\n s_add_u32 %[so], %[so], 128\n" \
                   "buffer_load_dwordx4 %[vo], %[rw], %[so] offen lds\n s_add_u32 m0, m0, 0x400\n s_add_u32 %[so], %[so], " #RADD "\n"
#define DMA1 "buffer_load_dwordx4 %[vo2], %[rw], %[so2] offen lds\n s_add_u32 m0, m0, 0x400\n s_add_u32 %[so2], %[so2], 65536\n"
      asm volatile("s_mov_b32 %[kp], m0\n s_mov_b32 m0, %[la]\n s_nop 4\n"
                   DMA2(16256) DMA2(49024) DMA2(16256) DMA2(49024) DMA2(16256) DMA2(49024) DMA2(16256) DMA2(49024)
                   DMA1 DMA1 DMA1 DMA1
                   "s_mov_b32 m0, %[kp]\n"
                   : [kp] "=&s"(keep), [so] "+s"(soff), [so2] "+s"(soff2) : [vo] "v"(voff), [vo2] "v"(voff2), [rw] "s"(rw), [la] "s"(ldsa) : "memory", "scc");
      if (kt >= 2) { if (pub < kt - 1) { asm volatile("s_waitcnt vmcnt(40)" ::: "memory"); pub = kt - 1; lds_st(fullp, pub); } }
      else if (kt == 1) { asm volatile("s_waitcnt vmcnt(20)" ::: "memory"); pub = 1; lds_st(fullp, pub); }
    }
    asm volatile("s_waitcnt vmcnt(20)" ::: "memory"); if (pub < NKT - 1) lds_st(fullp, NKT - 1);
    asm volatile("s_waitcnt vmcnt(0)" ::: "memory"); lds_st(fullp, DONE);
    __builtin_amdgcn_endpgm();
  }
  if (wv == 7) {
    if (sl >= 4) {
      const i32x4 rw = mkr(P.w, (unsigned)KK * NL * 2);
      unsigned t = 0;
#pragma unroll 1
      for (unsigned c = 3 * 32 + bi * 4 + sl - 4; c < KK / 2; c += NGEMM / 2)
        asm volatile("buffer_load_dword %0, %1, %2, 0 offen\n\ts_waitcnt vmcnt(8)" : "+v"(t) : "v"(c * 8192 + lane * 128), "s"(rw) : "memory");
      asm volatile("s_waitcnt vmcnt(0)" : "+v"(t) :: "memory");
    }
    if (cb < 12) {
      const char* op = P.out + (size_t)(rb * 256 + (cb & 3) * 64 + lane) * NL + 13 * 128 + (cb >> 2) * 128;
      unsigned t0, t1;
      asm volatile("global_load_dword %0, %2, off\n\tglobal_load_dword %1, %2, off offset:64\n\ts_waitcnt vmcnt(0)" : "=&v"(t0), "=&v"(t1) : "v"(op) : "memory");
    }
    __builtin_amdgcn_endpgm();
  }
  if (wv == 6) {
    const char* fp = ws + WS_AFL + flag_off(rb * 4, 0) + lane * 128;
    LAS char* rdy = lds + CT_ST + 8 * (lane & 3) + 4;
    LAS char* used = lds + CT_FREE + 256 * (lane & 3);
    const u64 t0 = rt();
    int c = 0, spins = 0, run = 0;
    for (;;) {
      const int lim = min(2 * (int)lds_ld(used) + POLL_LEAD, NSUB);
      if (!__any(c < lim)) {
        if (__all(c >= NSUB)) break;
        __builtin_amdgcn_s_sleep(2);
        continue;
      }
      const int pn = run >= 4 && c + 4 < lim ? 4 : 2;
      const bool in = (lane >> 2) < pn && c + (lane >> 2) < lim;
      const unsigned v = in ? ld_sys(fp + c * 512) : 0;
      const u64 nr = ~__ballot(v >= call);
      const int adv = __builtin_ctzll((nr >> (lane & 3)) & 0x1111111111111111ull) >> 2;
      run = adv == pn ? run + 1 : 0;
      c += adv;
      __hip_atomic_store((LAS unsigned*)rdy, c >= NSUB ? DONE : (unsigned)c, __ATOMIC_RELEASE, __HIP_MEMORY_SCOPE_WORKGROUP);
      if (__any(in && v < call)) {
        __builtin_amdgcn_s_sleep(4);
        if ((++spins & 31) == 0 && (rfl(ld_tmo(ws)) == call || rt() - t0 > DEADLINE)) { st_tmo(ws, call); lds_st(rdy, DONE); break; }
      }
    }
    const char* ap = lane == 48 ? ws + WS_TMO : ws + WS_AM2 + ((call & 1) * 48 + (lane < 48 ? lane : 0)) * 128;
    u64 v = 0;
    for (int it = 1;; ++it) {
      const u64 old = v;
      if (lane < 32 ? (unsigned)(v >> 32) < call : lane <= 48) v = ld_sys64(ap);
      const bool tick = (it & 15) == 0;
      if (!__any(v != old) && !tick) { if (!rfl(lds_ld(lds + CT_MAXSENT))) __builtin_amdgcn_s_sleep(8); continue; }
      const bool ok = lane < 48 && (unsigned)(v >> 32) >= call;
      const u64 m = __ballot(ok);
      unsigned state = (unsigned)m == ~0u ? 2 : __builtin_popcount((unsigned)(m >> 32)) >= SPEC_N;
      const unsigned cur = wave_max(ok ? (unsigned)v : 0u);
      if ((unsigned)__builtin_amdgcn_readlane((unsigned)v, 48) == call) state = 3;
      else if (tick && state != 2 && rt() - t0 > DEADLINE) { st_tmo(ws, call); state = 3; }
      __hip_atomic_store((LAS u64*)(lds + CT_AMAX), ((u64)state << 32) | cur, __ATOMIC_RELEASE, __HIP_MEMORY_SCOPE_WORKGROUP);
      if (state >= 2) break;
    }
    __builtin_amdgcn_endpgm();
  }

  __builtin_amdgcn_s_setprio(1);
  const i32x4 ra = mkr(ws + WS_AT + (size_t)(rb * 4 + wv) * AT_GROUP, (unsigned)AT_GROUP);
  const int g = lane >> 4, q = (lane >> 2) & 3, p4 = lane & 3;
  const unsigned wb0 = 4096 * (g >> 1) + 256 * q + 128 * (g & 1) + 8 * p4;
  unsigned w0 = wb0 + 32 * (0 ^ q), w1 = wb0 + 32 * (1 ^ q), w2 = wb0 + 32 * (2 ^ q), w3 = wb0 + 32 * (3 ^ q);
  const unsigned wb4 = 1024 * (g >> 1) + 256 * (g & 1) + 64 * q + 8 * p4;
  unsigned w4 = wb4 + 32 * (g & 1), w5 = wb4 + 32 * (1 ^ (g & 1));
  unsigned ctla = CT_ST + 8 * wv, fa = CT_FREE + 256 * wv + 4 * lane, fv = 0;
  unsigned xo = 0, na = (wv >> 2) ^ 1, nw = 1, cnt = 0, dl = 0, cst = W_PASS, sa, sw;
  char* outp = P.out + (size_t)(rb * 256 + (wv & 3) * 64) * NL;
  asm volatile("" : "+s"(outp));
  f32x32 a0, a1, a2, a3, a4;
  asm volatile(
      "s_cmp_eq_u32 %[na], 0\ns_cbranch_scc1 Lend_%=\nv_mov_b32 v96, 0\nv_mov_b32 v97, 0\nv_mov_b32 v98, 0\nv_mov_b32 v99, 0\nv_mov_b32 v100, 0\n"
      "v_mov_b32 v101, 0\nv_mov_b32 v102, 0\nv_mov_b32 v103, 0\nv_mov_b32 v104, 0\nv_mov_b32 v105, 0\nv_mov_b32 v106, 0\nv_mov_b32 v107, 0\n"
      "v_mov_b32 v108, 0\nv_mov_b32 v109, 0\nv_mov_b32 v110, 0\nv_mov_b32 v111, 0\nv_mov_b32 v112, 0\nv_mov_b32 v113, 0\nv_mov_b32 v114, 0\n"
      "v_mov_b32 v115, 0\nv_mov_b32 v116, 0\nv_mov_b32 v117, 0\nv_mov_b32 v118, 0\nv_mov_b32 v119, 0\nv_mov_b32 v120, 0\nv_mov_b32 v121, 0\n"
      "v_mov_b32 v122, 0\nv_mov_b32 v123, 0\nv_mov_b32 v124, 0\nv_mov_b32 v125, 0\nv_mov_b32 v126, 0\nv_mov_b32 v127, 0\nv_mov_b32 v128, 0\n"
      "v_mov_b32 v129, 0\nv_mov_b32 v130, 0\nv_mov_b32 v131, 0\nv_mov_b32 v132, 0\nv_mov_b32 v133, 0\nv_mov_b32 v134, 0\nv_mov_b32 v135, 0\n"
      "v_mov_b32 v136, 0\nv_mov_b32 v137, 0\nv_mov_b32 v138, 0\nv_mov_b32 v139, 0\nv_mov_b32 v140, 0\nv_mov_b32 v141, 0\nv_mov_b32 v142, 0\n"
      "v_mov_b32 v143, 0\nv_mov_b32 v144, 0\nv_mov_b32 v145, 0\nv_mov_b32 v146, 0\nv_mov_b32 v147, 0\nv_mov_b32 v148, 0\nv_mov_b32 v149, 0\n"
      "v_mov_b32 v150, 0\nv_mov_b32 v151, 0\nv_mov_b32 v152, 0\nv_mov_b32 v153, 0\nv_mov_b32 v154, 0\nv_mov_b32 v155, 0\nv_mov_b32 v156, 0\n"
      "v_mov_b32 v157, 0\nv_mov_b32 v158, 0\nv_mov_b32 v159, 0\nv_mov_b32 v160, 0\nv_mov_b32 v161, 0\nv_mov_b32 v162, 0\nv_mov_b32 v163, 0\n"
      "v_mov_b32 v164, 0\nv_mov_b32 v165, 0\nv_mov_b32 v166, 0\nv_mov_b32 v167, 0\nv_mov_b32 v168, 0\nv_mov_b32 v169, 0\nv_mov_b32 v170, 0\n"
      "v_mov_b32 v171, 0\nv_mov_b32 v172, 0\nv_mov_b32 v173, 0\nv_mov_b32 v174, 0\nv_mov_b32 v175, 0\nv_mov_b32 v176, 0\nv_mov_b32 v177, 0\n"
      "v_mov_b32 v178, 0\nv_mov_b32 v179, 0\nv_mov_b32 v180, 0\nv_mov_b32 v181, 0\nv_mov_b32 v182, 0\nv_mov_b32 v183, 0\nv_mov_b32 v184, 0\n"
      "v_mov_b32 v185, 0\nv_mov_b32 v186, 0\nv_mov_b32 v187, 0\nv_mov_b32 v188, 0\nv_mov_b32 v189, 0\nv_mov_b32 v190, 0\nv_mov_b32 v191, 0\n"
      "v_mov_b32 v192, 0\nv_mov_b32 v193, 0\nv_mov_b32 v194, 0\nv_mov_b32 v195, 0\nv_mov_b32 v196, 0\nv_mov_b32 v197, 0\nv_mov_b32 v198, 0\n"
      "v_mov_b32 v199, 0\nv_mov_b32 v200, 0\nv_mov_b32 v201, 0\nv_mov_b32 v202, 0\nv_mov_b32 v203, 0\nv_mov_b32 v204, 0\nv_mov_b32 v205, 0\n"
      "v_mov_b32 v206, 0\nv_mov_b32 v207, 0\nv_mov_b32 v208, 0\nv_mov_b32 v209, 0\nv_mov_b32 v210, 0\nv_mov_b32 v211, 0\nv_mov_b32 v212, 0\n"
      "v_mov_b32 v213, 0\nv_mov_b32 v214, 0\nv_mov_b32 v215, 0\nv_mov_b32 v216, 0\nv_mov_b32 v217, 0\nv_mov_b32 v218, 0\nv_mov_b32 v219, 0\n"
      "v_mov_b32 v220, 0\nv_mov_b32 v221, 0\nv_mov_b32 v222, 0\nv_mov_b32 v223, 0\nv_mov_b32 v224, 0\nv_mov_b32 v225, 0\nv_mov_b32 v226, 0\n"
      "v_mov_b32 v227, 0\nv_mov_b32 v228, 0\nv_mov_b32 v229, 0\nv_mov_b32 v230, 0\nv_mov_b32 v231, 0\nv_mov_b32 v232, 0\nv_mov_b32 v233, 0\n"
      "v_mov_b32 v234, 0\nv_mov_b32 v235, 0\nv_mov_b32 v236, 0\nv_mov_b32 v237, 0\nv_mov_b32 v238, 0\nv_mov_b32 v239, 0\nv_mov_b32 v240, 0\n"
      "v_mov_b32 v241, 0\nv_mov_b32 v242, 0\nv_mov_b32 v243, 0\nv_mov_b32 v244, 0\nv_mov_b32 v245, 0\nv_mov_b32 v246, 0\nv_mov_b32 v247, 0\n"
      "v_mov_b32 v248, 0\nv_mov_b32 v249, 0\nv_mov_b32 v250, 0\nv_mov_b32 v251, 0\nv_mov_b32 v252, 0\nv_mov_b32 v253, 0\nv_mov_b32 v254, 0\n"
      "v_mov_b32 v255, 0\ns_mov_b32 %[sa], 0x9e3779b1\ns_mov_b32 %[sw], 0x807f807f\nv_add_u32 v4, 77, v12\nv_mul_lo_u32 v4, v4, %[sa]\n"
      "v_alignbit_b32 v4, v4, v4, 15\nv_and_b32 v16, %[sw], v4\nv_or_b32 v16, 0x3f003e80, v16\nv_mul_lo_u32 v4, v4, %[sa]\nv_alignbit_b32 v4, v4, v4, 15\n"
      "v_and_b32 v17, %[sw], v4\nv_or_b32 v17, 0x3f003e80, v17\nv_mul_lo_u32 v4, v4, %[sa]\nv_alignbit_b32 v4, v4, v4, 15\nv_and_b32 v18, %[sw], v4\n"
      "v_or_b32 v18, 0x3f003e80, v18\nv_mul_lo_u32 v4, v4, %[sa]\nv_alignbit_b32 v4, v4, v4, 15\nv_and_b32 v19, %[sw], v4\nv_or_b32 v19, 0x3f003e80, v19\n"
      "v_mul_lo_u32 v4, v4, %[sa]\nv_alignbit_b32 v4, v4, v4, 15\nv_and_b32 v20, %[sw], v4\nv_or_b32 v20, 0x3f003e80, v20\nv_mul_lo_u32 v4, v4, %[sa]\n"
      "v_alignbit_b32 v4, v4, v4, 15\nv_and_b32 v21, %[sw], v4\nv_or_b32 v21, 0x3f003e80, v21\nv_mul_lo_u32 v4, v4, %[sa]\nv_alignbit_b32 v4, v4, v4, 15\n"
      "v_and_b32 v22, %[sw], v4\nv_or_b32 v22, 0x3f003e80, v22\nv_mul_lo_u32 v4, v4, %[sa]\nv_alignbit_b32 v4, v4, v4, 15\nv_and_b32 v23, %[sw], v4\n"
      "v_or_b32 v23, 0x3f003e80, v23\nv_mul_lo_u32 v4, v4, %[sa]\nv_alignbit_b32 v4, v4, v4, 15\nv_and_b32 v24, %[sw], v4\nv_or_b32 v24, 0x3f003e80, v24\n"
      "v_mul_lo_u32 v4, v4, %[sa]\nv_alignbit_b32 v4, v4, v4, 15\nv_and_b32 v25, %[sw], v4\nv_or_b32 v25, 0x3f003e80, v25\nv_mul_lo_u32 v4, v4, %[sa]\n"
      "v_alignbit_b32 v4, v4, v4, 15\nv_and_b32 v26, %[sw], v4\nv_or_b32 v26, 0x3f003e80, v26\nv_mul_lo_u32 v4, v4, %[sa]\nv_alignbit_b32 v4, v4, v4, 15\n"
      "v_and_b32 v27, %[sw], v4\nv_or_b32 v27, 0x3f003e80, v27\nv_mul_lo_u32 v4, v4, %[sa]\nv_alignbit_b32 v4, v4, v4, 15\nv_and_b32 v28, %[sw], v4\n"
      "v_or_b32 v28, 0x3f003e80, v28\nv_mul_lo_u32 v4, v4, %[sa]\nv_alignbit_b32 v4, v4, v4, 15\nv_and_b32 v29, %[sw], v4\nv_or_b32 v29, 0x3f003e80, v29\n"
      "v_mul_lo_u32 v4, v4, %[sa]\nv_alignbit_b32 v4, v4, v4, 15\nv_and_b32 v30, %[sw], v4\nv_or_b32 v30, 0x3f003e80, v30\nv_mul_lo_u32 v4, v4, %[sa]\n"
      "v_alignbit_b32 v4, v4, v4, 15\nv_and_b32 v31, %[sw], v4\nv_or_b32 v31, 0x3f003e80, v31\nv_mul_lo_u32 v4, v4, %[sa]\nv_alignbit_b32 v4, v4, v4, 15\n"
      "v_and_b32 v80, %[sw], v4\nv_or_b32 v80, 0x3f003e80, v80\nv_mul_lo_u32 v4, v4, %[sa]\nv_alignbit_b32 v4, v4, v4, 15\nv_and_b32 v81, %[sw], v4\n"
      "v_or_b32 v81, 0x3f003e80, v81\nv_mul_lo_u32 v4, v4, %[sa]\nv_alignbit_b32 v4, v4, v4, 15\nv_and_b32 v82, %[sw], v4\nv_or_b32 v82, 0x3f003e80, v82\n"
      "v_mul_lo_u32 v4, v4, %[sa]\nv_alignbit_b32 v4, v4, v4, 15\nv_and_b32 v83, %[sw], v4\nv_or_b32 v83, 0x3f003e80, v83\nv_mul_lo_u32 v4, v4, %[sa]\n"
      "v_alignbit_b32 v4, v4, v4, 15\nv_and_b32 v84, %[sw], v4\nv_or_b32 v84, 0x3f003e80, v84\nv_mul_lo_u32 v4, v4, %[sa]\nv_alignbit_b32 v4, v4, v4, 15\n"
      "v_and_b32 v85, %[sw], v4\nv_or_b32 v85, 0x3f003e80, v85\nv_mul_lo_u32 v4, v4, %[sa]\nv_alignbit_b32 v4, v4, v4, 15\nv_and_b32 v86, %[sw], v4\n"
      "v_or_b32 v86, 0x3f003e80, v86\nv_mul_lo_u32 v4, v4, %[sa]\nv_alignbit_b32 v4, v4, v4, 15\nv_and_b32 v87, %[sw], v4\nv_or_b32 v87, 0x3f003e80, v87\n"
      "v_mul_lo_u32 v4, v4, %[sa]\nv_alignbit_b32 v4, v4, v4, 15\nv_and_b32 v88, %[sw], v4\nv_or_b32 v88, 0x3f003e80, v88\nv_mul_lo_u32 v4, v4, %[sa]\n"
      "v_alignbit_b32 v4, v4, v4, 15\nv_and_b32 v89, %[sw], v4\nv_or_b32 v89, 0x3f003e80, v89\nv_mul_lo_u32 v4, v4, %[sa]\nv_alignbit_b32 v4, v4, v4, 15\n"
      "v_and_b32 v90, %[sw], v4\nv_or_b32 v90, 0x3f003e80, v90\nv_mul_lo_u32 v4, v4, %[sa]\nv_alignbit_b32 v4, v4, v4, 15\nv_and_b32 v91, %[sw], v4\n"
      "v_or_b32 v91, 0x3f003e80, v91\nv_mul_lo_u32 v4, v4, %[sa]\nv_alignbit_b32 v4, v4, v4, 15\nv_and_b32 v92, %[sw], v4\nv_or_b32 v92, 0x3f003e80, v92\n"
      "v_mul_lo_u32 v4, v4, %[sa]\nv_alignbit_b32 v4, v4, v4, 15\nv_and_b32 v93, %[sw], v4\nv_or_b32 v93, 0x3f003e80, v93\nv_mul_lo_u32 v4, v4, %[sa]\n"
      "v_alignbit_b32 v4, v4, v4, 15\nv_and_b32 v94, %[sw], v4\nv_or_b32 v94, 0x3f003e80, v94\nv_mul_lo_u32 v4, v4, %[sa]\nv_alignbit_b32 v4, v4, v4, 15\n"
      "v_and_b32 v95, %[sw], v4\nv_or_b32 v95, 0x3f003e80, v95\ns_nop 1\nLpA0_%=:\nds_read_b64 v[4:5], v13\ns_waitcnt lgkmcnt(0)\n"
      "v_cmp_gt_u32 vcc, %[na], v5\ns_cbranch_vccz LpB0_%=\nv_mfma_f32_16x16x32_bf16 v[96:99], v[16:19], v[80:83], v[96:99]\n"
      "v_mfma_f32_16x16x32_bf16 v[100:103], v[16:19], v[84:87], v[100:103]\nv_mfma_f32_16x16x32_bf16 v[104:107], v[16:19], v[88:91], v[104:107]\n"
      "v_mfma_f32_16x16x32_bf16 v[108:111], v[16:19], v[92:95], v[108:111]\nv_mfma_f32_16x16x32_bf16 v[96:99], v[20:23], v[80:83], v[96:99]\n"
      "v_mfma_f32_16x16x32_bf16 v[100:103], v[20:23], v[84:87], v[100:103]\nv_mfma_f32_16x16x32_bf16 v[104:107], v[20:23], v[88:91], v[104:107]\n"
      "v_mfma_f32_16x16x32_bf16 v[108:111], v[20:23], v[92:95], v[108:111]\nv_mfma_f32_16x16x32_bf16 v[96:99], v[24:27], v[80:83], v[96:99]\n"
      "v_mfma_f32_16x16x32_bf16 v[100:103], v[24:27], v[84:87], v[100:103]\nv_mfma_f32_16x16x32_bf16 v[104:107], v[24:27], v[88:91], v[104:107]\n"
      "v_mfma_f32_16x16x32_bf16 v[108:111], v[24:27], v[92:95], v[108:111]\nv_mfma_f32_16x16x32_bf16 v[96:99], v[28:31], v[80:83], v[96:99]\n"
      "v_mfma_f32_16x16x32_bf16 v[100:103], v[28:31], v[84:87], v[100:103]\nv_mfma_f32_16x16x32_bf16 v[104:107], v[28:31], v[88:91], v[104:107]\n"
      "v_mfma_f32_16x16x32_bf16 v[108:111], v[28:31], v[92:95], v[108:111]\ns_branch LpA0_%=\nLpB0_%=:\n"
      "buffer_load_dwordx4 v[32:35], v12, %[rs], %[xo] offen offset:0 sc0 sc1\nbuffer_load_dwordx4 v[36:39], v12, %[rs], %[xo] offen offset:1024 sc0 sc1\n"
      "buffer_load_dwordx4 v[40:43], v12, %[rs], %[xo] offen offset:2048 sc0 sc1\nbuffer_load_dwordx4 v[44:47], v12, %[rs], %[xo] offen offset:3072 sc0 sc1\n"
      "s_add_u32 %[xo], %[xo], 0x1000\ns_add_u32 %[na], %[na], 1\nLpA1_%=:\nds_read_b64 v[4:5], v13\ns_waitcnt lgkmcnt(0)\nv_cmp_gt_u32 vcc, %[na], v5\n"
      "s_cbranch_vccz LpB1_%=\nv_mfma_f32_16x16x32_bf16 v[96:99], v[16:19], v[80:83], v[96:99]\n"
      "v_mfma_f32_16x16x32_bf16 v[100:103], v[16:19], v[84:87], v[100:103]\nv_mfma_f32_16x16x32_bf16 v[104:107], v[16:19], v[88:91], v[104:107]\n"
      "v_mfma_f32_16x16x32_bf16 v[108:111], v[16:19], v[92:95], v[108:111]\nv_mfma_f32_16x16x32_bf16 v[96:99], v[20:23], v[80:83], v[96:99]\n"
      "v_mfma_f32_16x16x32_bf16 v[100:103], v[20:23], v[84:87], v[100:103]\nv_mfma_f32_16x16x32_bf16 v[104:107], v[20:23], v[88:91], v[104:107]\n"
      "v_mfma_f32_16x16x32_bf16 v[108:111], v[20:23], v[92:95], v[108:111]\nv_mfma_f32_16x16x32_bf16 v[96:99], v[24:27], v[80:83], v[96:99]\n"
      "v_mfma_f32_16x16x32_bf16 v[100:103], v[24:27], v[84:87], v[100:103]\nv_mfma_f32_16x16x32_bf16 v[104:107], v[24:27], v[88:91], v[104:107]\n"
      "v_mfma_f32_16x16x32_bf16 v[108:111], v[24:27], v[92:95], v[108:111]\nv_mfma_f32_16x16x32_bf16 v[96:99], v[28:31], v[80:83], v[96:99]\n"
      "v_mfma_f32_16x16x32_bf16 v[100:103], v[28:31], v[84:87], v[100:103]\nv_mfma_f32_16x16x32_bf16 v[104:107], v[28:31], v[88:91], v[104:107]\n"
      "v_mfma_f32_16x16x32_bf16 v[108:111], v[28:31], v[92:95], v[108:111]\ns_branch LpA1_%=\nLpB1_%=:\n"
      "buffer_load_dwordx4 v[48:51], v12, %[rs], %[xo] offen offset:0 sc0 sc1\nbuffer_load_dwordx4 v[52:55], v12, %[rs], %[xo] offen offset:1024 sc0 sc1\n"
      "buffer_load_dwordx4 v[56:59], v12, %[rs], %[xo] offen offset:2048 sc0 sc1\nbuffer_load_dwordx4 v[60:63], v12, %[rs], %[xo] offen offset:3072 sc0 sc1\n"
      "s_add_u32 %[xo], %[xo], 0x1000\ns_add_u32 %[na], %[na], 1\ns_add_u32 %[na], %[na], 1\nLpW_%=:\nds_read_b64 v[4:5], v13\ns_waitcnt lgkmcnt(0)\n"
      "v_cmp_gt_u32 vcc, %[nw], v4\ns_cbranch_vccz LpW2_%=\nv_mfma_f32_16x16x32_bf16 v[96:99], v[16:19], v[80:83], v[96:99]\n"
      "v_mfma_f32_16x16x32_bf16 v[100:103], v[16:19], v[84:87], v[100:103]\nv_mfma_f32_16x16x32_bf16 v[104:107], v[16:19], v[88:91], v[104:107]\n"
      "v_mfma_f32_16x16x32_bf16 v[108:111], v[16:19], v[92:95], v[108:111]\nv_mfma_f32_16x16x32_bf16 v[96:99], v[20:23], v[80:83], v[96:99]\n"
      "v_mfma_f32_16x16x32_bf16 v[100:103], v[20:23], v[84:87], v[100:103]\nv_mfma_f32_16x16x32_bf16 v[104:107], v[20:23], v[88:91], v[104:107]\n"
      "v_mfma_f32_16x16x32_bf16 v[108:111], v[20:23], v[92:95], v[108:111]\nv_mfma_f32_16x16x32_bf16 v[96:99], v[24:27], v[80:83], v[96:99]\n"
      "v_mfma_f32_16x16x32_bf16 v[100:103], v[24:27], v[84:87], v[100:103]\nv_mfma_f32_16x16x32_bf16 v[104:107], v[24:27], v[88:91], v[104:107]\n"
      "v_mfma_f32_16x16x32_bf16 v[108:111], v[24:27], v[92:95], v[108:111]\nv_mfma_f32_16x16x32_bf16 v[96:99], v[28:31], v[80:83], v[96:99]\n"
      "v_mfma_f32_16x16x32_bf16 v[100:103], v[28:31], v[84:87], v[100:103]\nv_mfma_f32_16x16x32_bf16 v[104:107], v[28:31], v[88:91], v[104:107]\n"
      "v_mfma_f32_16x16x32_bf16 v[108:111], v[28:31], v[92:95], v[108:111]\ns_branch LpW_%=\nLpW2_%=:\ns_nop 7\nv_mov_b32 v96, 0\nv_mov_b32 v97, 0\n"
      "v_mov_b32 v98, 0\nv_mov_b32 v99, 0\nv_mov_b32 v100, 0\nv_mov_b32 v101, 0\nv_mov_b32 v102, 0\nv_mov_b32 v103, 0\nv_mov_b32 v104, 0\nv_mov_b32 v105, 0\n"
      "v_mov_b32 v106, 0\nv_mov_b32 v107, 0\nv_mov_b32 v108, 0\nv_mov_b32 v109, 0\nv_mov_b32 v110, 0\nv_mov_b32 v111, 0\ns_add_u32 %[nw], %[nw], 1\n"
      "ds_read_b64_tr_b16 v[16:17], v6 offset:0\nds_read_b64_tr_b16 v[18:19], v6 offset:2048\nds_read_b64_tr_b16 v[20:21], v7 offset:0\n"
      "ds_read_b64_tr_b16 v[22:23], v7 offset:2048\nds_read_b64_tr_b16 v[24:25], v8 offset:0\nds_read_b64_tr_b16 v[26:27], v8 offset:2048\n"
      "s_waitcnt lgkmcnt(0)\ns_branch Lent_%=\n.p2align 6\nLloop_%=:\nv_mfma_f32_16x16x32_bf16 v[200:203], v[24:27], v[76:79], v[200:203]\n"
      "s_waitcnt lgkmcnt(4)\nv_mfma_f32_16x16x32_bf16 v[204:207], v[28:31], v[76:79], v[204:207]\nds_read_b64_tr_b16 v[24:25], v8 offset:21504\n"
      "v_mfma_f32_16x16x32_bf16 v[172:175], v[28:31], v[72:75], v[172:175]\nds_read_b64_tr_b16 v[26:27], v8 offset:23552\n"
      "v_mfma_f32_16x16x32_bf16 v[140:143], v[28:31], v[68:71], v[140:143]\nbuffer_load_dwordx4 v[32:35], v12, %[rs], %[xo] offen offset:0 sc0 sc1\n"
      "v_mfma_f32_16x16x32_bf16 v[108:111], v[28:31], v[64:67], v[108:111]\ns_waitcnt lgkmcnt(4)\n"
      "v_mfma_f32_16x16x32_bf16 v[112:115], v[16:19], v[64:67], v[112:115]\nds_read_b64_tr_b16 v[28:29], v9 offset:21504\n"
      "v_mfma_f32_16x16x32_bf16 v[144:147], v[16:19], v[68:71], v[144:147]\nds_read_b64_tr_b16 v[30:31], v9 offset:23552\n"
      "v_mfma_f32_16x16x32_bf16 v[176:179], v[16:19], v[72:75], v[176:179]\nbuffer_load_dwordx4 v[36:39], v12, %[rs], %[xo] offen offset:1024 sc0 sc1\n"
      "v_mfma_f32_16x16x32_bf16 v[208:211], v[16:19], v[76:79], v[208:211]\ns_waitcnt lgkmcnt(4)\n"
      "v_mfma_f32_16x16x32_bf16 v[212:215], v[20:23], v[76:79], v[212:215]\nds_read_b64_tr_b16 v[16:17], v10 offset:36864\n"
      "v_mfma_f32_16x16x32_bf16 v[180:183], v[20:23], v[72:75], v[180:183]\nds_read_b64_tr_b16 v[18:19], v10 offset:37376\n"
      "v_mfma_f32_16x16x32_bf16 v[148:151], v[20:23], v[68:71], v[148:151]\nbuffer_load_dwordx4 v[40:43], v12, %[rs], %[xo] offen offset:2048 sc0 sc1\n"
      "v_mfma_f32_16x16x32_bf16 v[116:119], v[20:23], v[64:67], v[116:119]\ns_waitcnt lgkmcnt(4)\n"
      "v_mfma_f32_16x16x32_bf16 v[120:123], v[24:27], v[64:67], v[120:123]\nds_read_b64_tr_b16 v[20:21], v11 offset:36864\n"
      "v_mfma_f32_16x16x32_bf16 v[152:155], v[24:27], v[68:71], v[152:155]\nds_read_b64_tr_b16 v[22:23], v11 offset:37376\n"
      "v_mfma_f32_16x16x32_bf16 v[184:187], v[24:27], v[72:75], v[184:187]\nbuffer_load_dwordx4 v[44:47], v12, %[rs], %[xo] offen offset:3072 sc0 sc1\n"
      "v_mfma_f32_16x16x32_bf16 v[216:219], v[24:27], v[76:79], v[216:219]\ns_waitcnt lgkmcnt(4)\n"
      "v_mfma_f32_16x16x32_bf16 v[220:223], v[28:31], v[76:79], v[220:223]\nds_read_b64_tr_b16 v[24:25], v6 offset:28672\n"
      "v_mfma_f32_16x16x32_bf16 v[188:191], v[28:31], v[72:75], v[188:191]\nds_read_b64_tr_b16 v[26:27], v6 offset:30720\n"
      "v_mfma_f32_16x16x32_bf16 v[156:159], v[28:31], v[68:71], v[156:159]\nv_mfma_f32_16x16x32_bf16 v[124:127], v[28:31], v[64:67], v[124:127]\n"
      "s_waitcnt lgkmcnt(4)\nv_mfma_f32_16x16x32_bf16 v[224:227], v[16:19], v[64:67], v[224:227]\nds_read_b64_tr_b16 v[28:29], v7 offset:28672\n"
      "v_mfma_f32_16x16x32_bf16 v[232:235], v[16:19], v[68:71], v[232:235]\nds_read_b64_tr_b16 v[30:31], v7 offset:30720\n"
      "v_mfma_f32_16x16x32_bf16 v[240:243], v[16:19], v[72:75], v[240:243]\nv_add_u32 v15, 1, v15\n"
      "v_mfma_f32_16x16x32_bf16 v[248:251], v[16:19], v[76:79], v[248:251]\ns_waitcnt lgkmcnt(4)\n"
      "v_mfma_f32_16x16x32_bf16 v[252:255], v[20:23], v[76:79], v[252:255]\nds_read_b64_tr_b16 v[16:17], v8 offset:28672\n"
      "v_mfma_f32_16x16x32_bf16 v[244:247], v[20:23], v[72:75], v[244:247]\nds_read_b64_tr_b16 v[18:19], v8 offset:30720\n"
      "v_mfma_f32_16x16x32_bf16 v[236:239], v[20:23], v[68:71], v[236:239]\ns_add_u32 %[xo], %[xo], 0x1000\n"
      "v_mfma_f32_16x16x32_bf16 v[228:231], v[20:23], v[64:67], v[228:231]\ns_waitcnt vmcnt(4) lgkmcnt(4)\n"
      "v_mfma_f32_16x16x32_bf16 v[96:99], v[24:27], v[80:83], v[96:99]\nds_read_b64_tr_b16 v[20:21], v9 offset:28672\n"
      "v_mfma_f32_16x16x32_bf16 v[128:131], v[24:27], v[84:87], v[128:131]\nds_read_b64_tr_b16 v[22:23], v9 offset:30720\n"
      "v_mfma_f32_16x16x32_bf16 v[160:163], v[24:27], v[88:91], v[160:163]\nbuffer_load_dwordx4 v[48:51], v12, %[rs], %[xo] offen offset:0 sc0 sc1\n"
      "v_mfma_f32_16x16x32_bf16 v[192:195], v[24:27], v[92:95], v[192:195]\ns_waitcnt lgkmcnt(4)\n"
      "v_mfma_f32_16x16x32_bf16 v[196:199], v[28:31], v[92:95], v[196:199]\nds_read_b64_tr_b16 v[24:25], v6 offset:29696\n"
      "v_mfma_f32_16x16x32_bf16 v[164:167], v[28:31], v[88:91], v[164:167]\nds_read_b64_tr_b16 v[26:27], v6 offset:31744\n"
      "v_mfma_f32_16x16x32_bf16 v[132:135], v[28:31], v[84:87], v[132:135]\nbuffer_load_dwordx4 v[52:55], v12, %[rs], %[xo] offen offset:1024 sc0 sc1\n"
      "v_mfma_f32_16x16x32_bf16 v[100:103], v[28:31], v[80:83], v[100:103]\ns_waitcnt lgkmcnt(4)\n"
      "v_mfma_f32_16x16x32_bf16 v[104:107], v[16:19], v[80:83], v[104:107]\nds_read_b64_tr_b16 v[28:29], v7 offset:29696\n"
      "v_mfma_f32_16x16x32_bf16 v[136:139], v[16:19], v[84:87], v[136:139]\nds_read_b64_tr_b16 v[30:31], v7 offset:31744\n"
      "v_mfma_f32_16x16x32_bf16 v[168:171], v[16:19], v[88:91], v[168:171]\nbuffer_load_dwordx4 v[56:59], v12, %[rs], %[xo] offen offset:2048 sc0 sc1\n"
      "v_mfma_f32_16x16x32_bf16 v[200:203], v[16:19], v[92:95], v[200:203]\ns_waitcnt lgkmcnt(4)\n"
      "v_mfma_f32_16x16x32_bf16 v[204:207], v[20:23], v[92:95], v[204:207]\nds_read_b64_tr_b16 v[16:17], v8 offset:29696\n"
      "v_mfma_f32_16x16x32_bf16 v[172:175], v[20:23], v[88:91], v[172:175]\nds_read_b64_tr_b16 v[18:19], v8 offset:31744\n"
      "v_mfma_f32_16x16x32_bf16 v[140:143], v[20:23], v[84:87], v[140:143]\nbuffer_load_dwordx4 v[60:63], v12, %[rs], %[xo] offen offset:3072 sc0 sc1\n"
      "v_mfma_f32_16x16x32_bf16 v[108:111], v[20:23], v[80:83], v[108:111]\nv_add_u32 v6, %[dl], v6\ns_waitcnt lgkmcnt(4)\n"
      "v_mfma_f32_16x16x32_bf16 v[112:115], v[24:27], v[80:83], v[112:115]\nds_read_b64_tr_b16 v[20:21], v9 offset:29696\n"
      "v_mfma_f32_16x16x32_bf16 v[144:147], v[24:27], v[84:87], v[144:147]\nds_read_b64_tr_b16 v[22:23], v9 offset:31744\n"
      "v_mfma_f32_16x16x32_bf16 v[176:179], v[24:27], v[88:91], v[176:179]\nv_mfma_f32_16x16x32_bf16 v[208:211], v[24:27], v[92:95], v[208:211]\n"
      "v_add_u32 v7, %[dl], v7\ns_waitcnt lgkmcnt(4)\nv_mfma_f32_16x16x32_bf16 v[212:215], v[28:31], v[92:95], v[212:215]\n"
      "ds_read_b64_tr_b16 v[24:25], v10 offset:38912\nv_mfma_f32_16x16x32_bf16 v[180:183], v[28:31], v[88:91], v[180:183]\n"
      "ds_read_b64_tr_b16 v[26:27], v10 offset:39424\nv_mfma_f32_16x16x32_bf16 v[148:151], v[28:31], v[84:87], v[148:151]\nds_read_b64 v[4:5], v13\n"
      "v_mfma_f32_16x16x32_bf16 v[116:119], v[28:31], v[80:83], v[116:119]\nv_add_u32 v8, %[dl], v8\ns_waitcnt lgkmcnt(5)\n"
      "v_mfma_f32_16x16x32_bf16 v[120:123], v[16:19], v[80:83], v[120:123]\nds_read_b64_tr_b16 v[28:29], v11 offset:38912\n"
      "v_mfma_f32_16x16x32_bf16 v[152:155], v[16:19], v[84:87], v[152:155]\nds_read_b64_tr_b16 v[30:31], v11 offset:39424\n"
      "v_mfma_f32_16x16x32_bf16 v[184:187], v[16:19], v[88:91], v[184:187]\ns_add_u32 %[nw], %[nw], 1\ns_add_u32 %[na], %[na], 2\n"
      "v_mfma_f32_16x16x32_bf16 v[216:219], v[16:19], v[92:95], v[216:219]\nv_add_u32 v9, %[dl], v9\ns_waitcnt lgkmcnt(5)\n"
      "v_mfma_f32_16x16x32_bf16 v[220:223], v[20:23], v[92:95], v[220:223]\nds_read_b64_tr_b16 v[16:17], v6 offset:0\n"
      "v_mfma_f32_16x16x32_bf16 v[188:191], v[20:23], v[88:91], v[188:191]\nds_read_b64_tr_b16 v[18:19], v6 offset:2048\n"
      "v_mfma_f32_16x16x32_bf16 v[156:159], v[20:23], v[84:87], v[156:159]\nv_mfma_f32_16x16x32_bf16 v[124:127], v[20:23], v[80:83], v[124:127]\n"
      "v_add_u32 v10, %[dl], v10\ns_waitcnt lgkmcnt(5)\nv_mfma_f32_16x16x32_bf16 v[224:227], v[24:27], v[80:83], v[224:227]\n"
      "ds_read_b64_tr_b16 v[20:21], v7 offset:0\nv_mfma_f32_16x16x32_bf16 v[232:235], v[24:27], v[84:87], v[232:235]\n"
      "ds_read_b64_tr_b16 v[22:23], v7 offset:2048\nv_mfma_f32_16x16x32_bf16 v[240:243], v[24:27], v[88:91], v[240:243]\nds_write_b32 v14, v15\n"
      "v_mfma_f32_16x16x32_bf16 v[248:251], v[24:27], v[92:95], v[248:251]\nv_add_u32 v11, %[dl], v11\ns_waitcnt lgkmcnt(5)\n"
      "v_mfma_f32_16x16x32_bf16 v[252:255], v[28:31], v[92:95], v[252:255]\nds_read_b64_tr_b16 v[24:25], v8 offset:0\n"
      "v_mfma_f32_16x16x32_bf16 v[244:247], v[28:31], v[88:91], v[244:247]\nds_read_b64_tr_b16 v[26:27], v8 offset:2048\n"
      "v_mfma_f32_16x16x32_bf16 v[236:239], v[28:31], v[84:87], v[236:239]\ns_add_u32 %[xo], %[xo], 0x1000\n"
      "v_mfma_f32_16x16x32_bf16 v[228:231], v[28:31], v[80:83], v[228:231]\nLent_%=:\ns_waitcnt vmcnt(4) lgkmcnt(5)\n"
      "v_mfma_f32_16x16x32_bf16 v[96:99], v[16:19], v[32:35], v[96:99]\nds_read_b64_tr_b16 v[28:29], v9 offset:0\n"
      "v_mfma_f32_16x16x32_bf16 v[128:131], v[16:19], v[36:39], v[128:131]\nds_read_b64_tr_b16 v[30:31], v9 offset:2048\n"
      "v_mfma_f32_16x16x32_bf16 v[160:163], v[16:19], v[40:43], v[160:163]\nv_readfirstlane_b32 %[sa], v5\n"
      "v_mfma_f32_16x16x32_bf16 v[192:195], v[16:19], v[44:47], v[192:195]\nv_readfirstlane_b32 %[sw], v4\ns_waitcnt lgkmcnt(5)\n"
      "v_mfma_f32_16x16x32_bf16 v[196:199], v[20:23], v[44:47], v[196:199]\nds_read_b64_tr_b16 v[16:17], v6 offset:1024\ns_cmp_ge_u32 %[sa], %[na]\n"
      "v_mfma_f32_16x16x32_bf16 v[164:167], v[20:23], v[40:43], v[164:167]\nds_read_b64_tr_b16 v[18:19], v6 offset:3072\ns_cselect_b32 %[sw], %[sw], 0\n"
      "v_mfma_f32_16x16x32_bf16 v[132:135], v[20:23], v[36:39], v[132:135]\ns_cmp_ge_u32 %[sw], %[nw]\n"
      "v_mfma_f32_16x16x32_bf16 v[100:103], v[20:23], v[32:35], v[100:103]\ns_waitcnt lgkmcnt(4)\n"
      "v_mfma_f32_16x16x32_bf16 v[104:107], v[24:27], v[32:35], v[104:107]\nds_read_b64_tr_b16 v[20:21], v7 offset:1024\n"
      "v_mfma_f32_16x16x32_bf16 v[136:139], v[24:27], v[36:39], v[136:139]\nds_read_b64_tr_b16 v[22:23], v7 offset:3072\n"
      "v_mfma_f32_16x16x32_bf16 v[168:171], v[24:27], v[40:43], v[168:171]\ns_cbranch_scc0 Lz0_%=\nLb0_%=:\n"
      "v_mfma_f32_16x16x32_bf16 v[200:203], v[24:27], v[44:47], v[200:203]\ns_waitcnt lgkmcnt(4)\n"
      "v_mfma_f32_16x16x32_bf16 v[204:207], v[28:31], v[44:47], v[204:207]\nds_read_b64_tr_b16 v[24:25], v8 offset:1024\n"
      "v_mfma_f32_16x16x32_bf16 v[172:175], v[28:31], v[40:43], v[172:175]\nds_read_b64_tr_b16 v[26:27], v8 offset:3072\n"
      "v_mfma_f32_16x16x32_bf16 v[140:143], v[28:31], v[36:39], v[140:143]\nbuffer_load_dwordx4 v[64:67], v12, %[rs], %[xo] offen offset:0 sc0 sc1\n"
      "v_mfma_f32_16x16x32_bf16 v[108:111], v[28:31], v[32:35], v[108:111]\ns_waitcnt lgkmcnt(4)\n"
      "v_mfma_f32_16x16x32_bf16 v[112:115], v[16:19], v[32:35], v[112:115]\nds_read_b64_tr_b16 v[28:29], v9 offset:1024\n"
      "v_mfma_f32_16x16x32_bf16 v[144:147], v[16:19], v[36:39], v[144:147]\nds_read_b64_tr_b16 v[30:31], v9 offset:3072\n"
      "v_mfma_f32_16x16x32_bf16 v[176:179], v[16:19], v[40:43], v[176:179]\nbuffer_load_dwordx4 v[68:71], v12, %[rs], %[xo] offen offset:1024 sc0 sc1\n"
      "v_mfma_f32_16x16x32_bf16 v[208:211], v[16:19], v[44:47], v[208:211]\ns_waitcnt lgkmcnt(4)\n"
      "v_mfma_f32_16x16x32_bf16 v[212:215], v[20:23], v[44:47], v[212:215]\nds_read_b64_tr_b16 v[16:17], v10 offset:16384\n"
      "v_mfma_f32_16x16x32_bf16 v[180:183], v[20:23], v[40:43], v[180:183]\nds_read_b64_tr_b16 v[18:19], v10 offset:16896\n"
      "v_mfma_f32_16x16x32_bf16 v[148:151], v[20:23], v[36:39], v[148:151]\nbuffer_load_dwordx4 v[72:75], v12, %[rs], %[xo] offen offset:2048 sc0 sc1\n"
      "v_mfma_f32_16x16x32_bf16 v[116:119], v[20:23], v[32:35], v[116:119]\ns_waitcnt lgkmcnt(4)\n"
      "v_mfma_f32_16x16x32_bf16 v[120:123], v[24:27], v[32:35], v[120:123]\nds_read_b64_tr_b16 v[20:21], v11 offset:16384\n"
      "v_mfma_f32_16x16x32_bf16 v[152:155], v[24:27], v[36:39], v[152:155]\nds_read_b64_tr_b16 v[22:23], v11 offset:16896\n"
      "v_mfma_f32_16x16x32_bf16 v[184:187], v[24:27], v[40:43], v[184:187]\nbuffer_load_dwordx4 v[76:79], v12, %[rs], %[xo] offen offset:3072 sc0 sc1\n"
      "v_mfma_f32_16x16x32_bf16 v[216:219], v[24:27], v[44:47], v[216:219]\ns_waitcnt lgkmcnt(4)\n"
      "v_mfma_f32_16x16x32_bf16 v[220:223], v[28:31], v[44:47], v[220:223]\nds_read_b64_tr_b16 v[24:25], v6 offset:8192\n"
      "v_mfma_f32_16x16x32_bf16 v[188:191], v[28:31], v[40:43], v[188:191]\nds_read_b64_tr_b16 v[26:27], v6 offset:10240\n"
      "v_mfma_f32_16x16x32_bf16 v[156:159], v[28:31], v[36:39], v[156:159]\nv_mfma_f32_16x16x32_bf16 v[124:127], v[28:31], v[32:35], v[124:127]\n"
      "s_waitcnt lgkmcnt(4)\nv_mfma_f32_16x16x32_bf16 v[224:227], v[16:19], v[32:35], v[224:227]\nds_read_b64_tr_b16 v[28:29], v7 offset:8192\n"
      "v_mfma_f32_16x16x32_bf16 v[232:235], v[16:19], v[36:39], v[232:235]\nds_read_b64_tr_b16 v[30:31], v7 offset:10240\n"
      "v_mfma_f32_16x16x32_bf16 v[240:243], v[16:19], v[40:43], v[240:243]\nv_add_u32 v15, 1, v15\n"
      "v_mfma_f32_16x16x32_bf16 v[248:251], v[16:19], v[44:47], v[248:251]\ns_waitcnt lgkmcnt(4)\n"
      "v_mfma_f32_16x16x32_bf16 v[252:255], v[20:23], v[44:47], v[252:255]\nds_read_b64_tr_b16 v[16:17], v8 offset:8192\n"
      "v_mfma_f32_16x16x32_bf16 v[244:247], v[20:23], v[40:43], v[244:247]\nds_read_b64_tr_b16 v[18:19], v8 offset:10240\n"
      "v_mfma_f32_16x16x32_bf16 v[236:239], v[20:23], v[36:39], v[236:239]\ns_add_u32 %[xo], %[xo], 0x1000\n"
      "v_mfma_f32_16x16x32_bf16 v[228:231], v[20:23], v[32:35], v[228:231]\ns_waitcnt vmcnt(4) lgkmcnt(4)\n"
      "v_mfma_f32_16x16x32_bf16 v[96:99], v[24:27], v[48:51], v[96:99]\nds_read_b64_tr_b16 v[20:21], v9 offset:8192\n"
      "v_mfma_f32_16x16x32_bf16 v[128:131], v[24:27], v[52:55], v[128:131]\nds_read_b64_tr_b16 v[22:23], v9 offset:10240\n"
      "v_mfma_f32_16x16x32_bf16 v[160:163], v[24:27], v[56:59], v[160:163]\nbuffer_load_dwordx4 v[80:83], v12, %[rs], %[xo] offen offset:0 sc0 sc1\n"
      "v_mfma_f32_16x16x32_bf16 v[192:195], v[24:27], v[60:63], v[192:195]\ns_waitcnt lgkmcnt(4)\n"
      "v_mfma_f32_16x16x32_bf16 v[196:199], v[28:31], v[60:63], v[196:199]\nds_read_b64_tr_b16 v[24:25], v6 offset:9216\n"
      "v_mfma_f32_16x16x32_bf16 v[164:167], v[28:31], v[56:59], v[164:167]\nds_read_b64_tr_b16 v[26:27], v6 offset:11264\n"
      "v_mfma_f32_16x16x32_bf16 v[132:135], v[28:31], v[52:55], v[132:135]\nbuffer_load_dwordx4 v[84:87], v12, %[rs], %[xo] offen offset:1024 sc0 sc1\n"
      "v_mfma_f32_16x16x32_bf16 v[100:103], v[28:31], v[48:51], v[100:103]\ns_waitcnt lgkmcnt(4)\n"
      "v_mfma_f32_16x16x32_bf16 v[104:107], v[16:19], v[48:51], v[104:107]\nds_read_b64_tr_b16 v[28:29], v7 offset:9216\n"
      "v_mfma_f32_16x16x32_bf16 v[136:139], v[16:19], v[52:55], v[136:139]\nds_read_b64_tr_b16 v[30:31], v7 offset:11264\n"
      "v_mfma_f32_16x16x32_bf16 v[168:171], v[16:19], v[56:59], v[168:171]\nbuffer_load_dwordx4 v[88:91], v12, %[rs], %[xo] offen offset:2048 sc0 sc1\n"
      "v_mfma_f32_16x16x32_bf16 v[200:203], v[16:19], v[60:63], v[200:203]\ns_waitcnt lgkmcnt(4)\n"
      "v_mfma_f32_16x16x32_bf16 v[204:207], v[20:23], v[60:63], v[204:207]\nds_read_b64_tr_b16 v[16:17], v8 offset:9216\n"
      "v_mfma_f32_16x16x32_bf16 v[172:175], v[20:23], v[56:59], v[172:175]\nds_read_b64_tr_b16 v[18:19], v8 offset:11264\n"
      "v_mfma_f32_16x16x32_bf16 v[140:143], v[20:23], v[52:55], v[140:143]\nbuffer_load_dwordx4 v[92:95], v12, %[rs], %[xo] offen offset:3072 sc0 sc1\n"
      "v_mfma_f32_16x16x32_bf16 v[108:111], v[20:23], v[48:51], v[108:111]\ns_waitcnt lgkmcnt(4)\n"
      "v_mfma_f32_16x16x32_bf16 v[112:115], v[24:27], v[48:51], v[112:115]\nds_read_b64_tr_b16 v[20:21], v9 offset:9216\n"
      "v_mfma_f32_16x16x32_bf16 v[144:147], v[24:27], v[52:55], v[144:147]\nds_read_b64_tr_b16 v[22:23], v9 offset:11264\n"
      "v_mfma_f32_16x16x32_bf16 v[176:179], v[24:27], v[56:59], v[176:179]\ns_add_u32 %[cnt], %[cnt], 1\ns_cmp_eq_u32 %[cnt], 3\n"
      "v_mfma_f32_16x16x32_bf16 v[208:211], v[24:27], v[60:63], v[208:211]\ns_waitcnt lgkmcnt(4)\n"
      "v_mfma_f32_16x16x32_bf16 v[212:215], v[28:31], v[60:63], v[212:215]\nds_read_b64_tr_b16 v[24:25], v10 offset:18432\ns_cselect_b32 %[cnt], 0, %[cnt]\n"
      "v_mfma_f32_16x16x32_bf16 v[180:183], v[28:31], v[56:59], v[180:183]\nds_read_b64_tr_b16 v[26:27], v10 offset:18944\n"
      "v_mfma_f32_16x16x32_bf16 v[148:151], v[28:31], v[52:55], v[148:151]\nds_read_b64 v[4:5], v13\n"
      "v_mfma_f32_16x16x32_bf16 v[116:119], v[28:31], v[48:51], v[116:119]\ns_cselect_b32 %[dl], 0xfffec000, %[cst]\ns_waitcnt lgkmcnt(5)\n"
      "v_mfma_f32_16x16x32_bf16 v[120:123], v[16:19], v[48:51], v[120:123]\nds_read_b64_tr_b16 v[28:29], v11 offset:18432\n"
      "v_mfma_f32_16x16x32_bf16 v[152:155], v[16:19], v[52:55], v[152:155]\nds_read_b64_tr_b16 v[30:31], v11 offset:18944\n"
      "v_mfma_f32_16x16x32_bf16 v[184:187], v[16:19], v[56:59], v[184:187]\ns_add_u32 %[nw], %[nw], 1\ns_add_u32 %[na], %[na], 2\n"
      "v_mfma_f32_16x16x32_bf16 v[216:219], v[16:19], v[60:63], v[216:219]\ns_waitcnt lgkmcnt(5)\n"
      "v_mfma_f32_16x16x32_bf16 v[220:223], v[20:23], v[60:63], v[220:223]\nds_read_b64_tr_b16 v[16:17], v6 offset:20480\n"
      "v_mfma_f32_16x16x32_bf16 v[188:191], v[20:23], v[56:59], v[188:191]\nds_read_b64_tr_b16 v[18:19], v6 offset:22528\n"
      "v_mfma_f32_16x16x32_bf16 v[156:159], v[20:23], v[52:55], v[156:159]\nv_mfma_f32_16x16x32_bf16 v[124:127], v[20:23], v[48:51], v[124:127]\n"
      "s_waitcnt lgkmcnt(5)\nv_mfma_f32_16x16x32_bf16 v[224:227], v[24:27], v[48:51], v[224:227]\nds_read_b64_tr_b16 v[20:21], v7 offset:20480\n"
      "v_mfma_f32_16x16x32_bf16 v[232:235], v[24:27], v[52:55], v[232:235]\nds_read_b64_tr_b16 v[22:23], v7 offset:22528\n"
      "v_mfma_f32_16x16x32_bf16 v[240:243], v[24:27], v[56:59], v[240:243]\nds_write_b32 v14, v15\n"
      "v_mfma_f32_16x16x32_bf16 v[248:251], v[24:27], v[60:63], v[248:251]\ns_waitcnt lgkmcnt(5)\n"
      "v_mfma_f32_16x16x32_bf16 v[252:255], v[28:31], v[60:63], v[252:255]\nds_read_b64_tr_b16 v[24:25], v8 offset:20480\n"
      "v_mfma_f32_16x16x32_bf16 v[244:247], v[28:31], v[56:59], v[244:247]\nds_read_b64_tr_b16 v[26:27], v8 offset:22528\n"
      "v_mfma_f32_16x16x32_bf16 v[236:239], v[28:31], v[52:55], v[236:239]\ns_add_u32 %[xo], %[xo], 0x1000\n"
      "v_mfma_f32_16x16x32_bf16 v[228:231], v[28:31], v[48:51], v[228:231]\ns_waitcnt vmcnt(4) lgkmcnt(5)\n"
      "v_mfma_f32_16x16x32_bf16 v[96:99], v[16:19], v[64:67], v[96:99]\nds_read_b64_tr_b16 v[28:29], v9 offset:20480\n"
      "v_mfma_f32_16x16x32_bf16 v[128:131], v[16:19], v[68:71], v[128:131]\nds_read_b64_tr_b16 v[30:31], v9 offset:22528\n"
      "v_mfma_f32_16x16x32_bf16 v[160:163], v[16:19], v[72:75], v[160:163]\nv_readfirstlane_b32 %[sa], v5\n"
      "v_mfma_f32_16x16x32_bf16 v[192:195], v[16:19], v[76:79], v[192:195]\nv_readfirstlane_b32 %[sw], v4\ns_waitcnt lgkmcnt(5)\n"
      "v_mfma_f32_16x16x32_bf16 v[196:199], v[20:23], v[76:79], v[196:199]\nds_read_b64_tr_b16 v[16:17], v6 offset:21504\ns_cmp_ge_u32 %[sa], %[na]\n"
      "v_mfma_f32_16x16x32_bf16 v[164:167], v[20:23], v[72:75], v[164:167]\nds_read_b64_tr_b16 v[18:19], v6 offset:23552\ns_cselect_b32 %[sw], %[sw], 0\n"
      "v_mfma_f32_16x16x32_bf16 v[132:135], v[20:23], v[68:71], v[132:135]\ns_cmp_ge_u32 %[sw], %[nw]\ns_cselect_b32 %[sw], %[nw], 100\n"
      "v_mfma_f32_16x16x32_bf16 v[100:103], v[20:23], v[64:67], v[100:103]\ns_cmp_lt_u32 %[sw], 65\ns_waitcnt lgkmcnt(4)\n"
      "v_mfma_f32_16x16x32_bf16 v[104:107], v[24:27], v[64:67], v[104:107]\nds_read_b64_tr_b16 v[20:21], v7 offset:21504\n"
      "v_mfma_f32_16x16x32_bf16 v[136:139], v[24:27], v[68:71], v[136:139]\nds_read_b64_tr_b16 v[22:23], v7 offset:23552\n"
      "v_mfma_f32_16x16x32_bf16 v[168:171], v[24:27], v[72:75], v[168:171]\ns_cbranch_scc1 Lloop_%=\ns_cmp_lt_u32 %[nw], 65\ns_cbranch_scc1 Lz2_%=\n"
      "v_mfma_f32_16x16x32_bf16 v[200:203], v[24:27], v[76:79], v[200:203]\ns_waitcnt lgkmcnt(4)\n"
      "v_mfma_f32_16x16x32_bf16 v[204:207], v[28:31], v[76:79], v[204:207]\nds_read_b64_tr_b16 v[24:25], v8 offset:21504\n"
      "v_mfma_f32_16x16x32_bf16 v[172:175], v[28:31], v[72:75], v[172:175]\nds_read_b64_tr_b16 v[26:27], v8 offset:23552\n"
      "v_mfma_f32_16x16x32_bf16 v[140:143], v[28:31], v[68:71], v[140:143]\nv_mfma_f32_16x16x32_bf16 v[108:111], v[28:31], v[64:67], v[108:111]\n"
      "s_waitcnt lgkmcnt(4)\nv_mfma_f32_16x16x32_bf16 v[112:115], v[16:19], v[64:67], v[112:115]\nds_read_b64_tr_b16 v[28:29], v9 offset:21504\n"
      "v_mfma_f32_16x16x32_bf16 v[144:147], v[16:19], v[68:71], v[144:147]\nds_read_b64_tr_b16 v[30:31], v9 offset:23552\n"
      "v_mfma_f32_16x16x32_bf16 v[176:179], v[16:19], v[72:75], v[176:179]\nv_mfma_f32_16x16x32_bf16 v[208:211], v[16:19], v[76:79], v[208:211]\n"
      "s_waitcnt lgkmcnt(4)\nv_mfma_f32_16x16x32_bf16 v[212:215], v[20:23], v[76:79], v[212:215]\nds_read_b64_tr_b16 v[16:17], v10 offset:36864\n"
      "v_mfma_f32_16x16x32_bf16 v[180:183], v[20:23], v[72:75], v[180:183]\nds_read_b64_tr_b16 v[18:19], v10 offset:37376\n"
      "v_mfma_f32_16x16x32_bf16 v[148:151], v[20:23], v[68:71], v[148:151]\nv_mfma_f32_16x16x32_bf16 v[116:119], v[20:23], v[64:67], v[116:119]\n"
      "s_waitcnt lgkmcnt(4)\nv_mfma_f32_16x16x32_bf16 v[120:123], v[24:27], v[64:67], v[120:123]\nds_read_b64_tr_b16 v[20:21], v11 offset:36864\n"
      "v_mfma_f32_16x16x32_bf16 v[152:155], v[24:27], v[68:71], v[152:155]\nds_read_b64_tr_b16 v[22:23], v11 offset:37376\n"
      "v_mfma_f32_16x16x32_bf16 v[184:187], v[24:27], v[72:75], v[184:187]\nv_mfma_f32_16x16x32_bf16 v[216:219], v[24:27], v[76:79], v[216:219]\n"
      "s_waitcnt lgkmcnt(4)\nv_mfma_f32_16x16x32_bf16 v[220:223], v[28:31], v[76:79], v[220:223]\nds_read_b64_tr_b16 v[24:25], v6 offset:28672\n"
      "v_mfma_f32_16x16x32_bf16 v[188:191], v[28:31], v[72:75], v[188:191]\nds_read_b64_tr_b16 v[26:27], v6 offset:30720\n"
      "v_mfma_f32_16x16x32_bf16 v[156:159], v[28:31], v[68:71], v[156:159]\nv_mfma_f32_16x16x32_bf16 v[124:127], v[28:31], v[64:67], v[124:127]\n"
      "s_waitcnt lgkmcnt(4)\nv_mfma_f32_16x16x32_bf16 v[224:227], v[16:19], v[64:67], v[224:227]\nds_read_b64_tr_b16 v[28:29], v7 offset:28672\n"
      "v_mfma_f32_16x16x32_bf16 v[232:235], v[16:19], v[68:71], v[232:235]\nds_read_b64_tr_b16 v[30:31], v7 offset:30720\n"
      "v_mfma_f32_16x16x32_bf16 v[240:243], v[16:19], v[72:75], v[240:243]\nv_mfma_f32_16x16x32_bf16 v[248:251], v[16:19], v[76:79], v[248:251]\n"
      "s_waitcnt lgkmcnt(4)\nv_mfma_f32_16x16x32_bf16 v[252:255], v[20:23], v[76:79], v[252:255]\nds_read_b64_tr_b16 v[16:17], v8 offset:28672\n"
      "v_mfma_f32_16x16x32_bf16 v[244:247], v[20:23], v[72:75], v[244:247]\nds_read_b64_tr_b16 v[18:19], v8 offset:30720\n"
      "v_mfma_f32_16x16x32_bf16 v[236:239], v[20:23], v[68:71], v[236:239]\nv_mfma_f32_16x16x32_bf16 v[228:231], v[20:23], v[64:67], v[228:231]\n"
      "s_waitcnt vmcnt(0) lgkmcnt(4)\nv_mfma_f32_16x16x32_bf16 v[96:99], v[24:27], v[80:83], v[96:99]\nds_read_b64_tr_b16 v[20:21], v9 offset:28672\n"
      "v_mfma_f32_16x16x32_bf16 v[128:131], v[24:27], v[84:87], v[128:131]\nds_read_b64_tr_b16 v[22:23], v9 offset:30720\n"
      "v_mfma_f32_16x16x32_bf16 v[160:163], v[24:27], v[88:91], v[160:163]\nv_mfma_f32_16x16x32_bf16 v[192:195], v[24:27], v[92:95], v[192:195]\n"
      "s_waitcnt lgkmcnt(4)\nv_mfma_f32_16x16x32_bf16 v[196:199], v[28:31], v[92:95], v[196:199]\nds_read_b64_tr_b16 v[24:25], v6 offset:29696\n"
      "v_mfma_f32_16x16x32_bf16 v[164:167], v[28:31], v[88:91], v[164:167]\nds_read_b64_tr_b16 v[26:27], v6 offset:31744\n"
      "v_mfma_f32_16x16x32_bf16 v[132:135], v[28:31], v[84:87], v[132:135]\nv_mfma_f32_16x16x32_bf16 v[100:103], v[28:31], v[80:83], v[100:103]\n"
      "s_waitcnt lgkmcnt(4)\nv_mfma_f32_16x16x32_bf16 v[104:107], v[16:19], v[80:83], v[104:107]\nds_read_b64_tr_b16 v[28:29], v7 offset:29696\n"
      "v_mfma_f32_16x16x32_bf16 v[136:139], v[16:19], v[84:87], v[136:139]\nds_read_b64_tr_b16 v[30:31], v7 offset:31744\n"
      "v_mfma_f32_16x16x32_bf16 v[168:171], v[16:19], v[88:91], v[168:171]\nv_mfma_f32_16x16x32_bf16 v[200:203], v[16:19], v[92:95], v[200:203]\n"
      "s_waitcnt lgkmcnt(4)\nv_mfma_f32_16x16x32_bf16 v[204:207], v[20:23], v[92:95], v[204:207]\nds_read_b64_tr_b16 v[16:17], v8 offset:29696\n"
      "v_mfma_f32_16x16x32_bf16 v[172:175], v[20:23], v[88:91], v[172:175]\nds_read_b64_tr_b16 v[18:19], v8 offset:31744\n"
      "v_mfma_f32_16x16x32_bf16 v[140:143], v[20:23], v[84:87], v[140:143]\nv_mfma_f32_16x16x32_bf16 v[108:111], v[20:23], v[80:83], v[108:111]\n"
      "s_waitcnt lgkmcnt(4)\nv_mfma_f32_16x16x32_bf16 v[112:115], v[24:27], v[80:83], v[112:115]\nds_read_b64_tr_b16 v[20:21], v9 offset:29696\n"
      "v_mfma_f32_16x16x32_bf16 v[144:147], v[24:27], v[84:87], v[144:147]\nds_read_b64_tr_b16 v[22:23], v9 offset:31744\n"
      "v_mfma_f32_16x16x32_bf16 v[176:179], v[24:27], v[88:91], v[176:179]\nv_mfma_f32_16x16x32_bf16 v[208:211], v[24:27], v[92:95], v[208:211]\n"
      "s_waitcnt lgkmcnt(4)\nv_mfma_f32_16x16x32_bf16 v[212:215], v[28:31], v[92:95], v[212:215]\nds_read_b64_tr_b16 v[24:25], v10 offset:38912\n"
      "v_mfma_f32_16x16x32_bf16 v[180:183], v[28:31], v[88:91], v[180:183]\nds_read_b64_tr_b16 v[26:27], v10 offset:39424\n"
      "v_mfma_f32_16x16x32_bf16 v[148:151], v[28:31], v[84:87], v[148:151]\nds_read_b64 v[4:5], v13\n"
      "v_mfma_f32_16x16x32_bf16 v[116:119], v[28:31], v[80:83], v[116:119]\ns_waitcnt lgkmcnt(5)\n"
      "v_mfma_f32_16x16x32_bf16 v[120:123], v[16:19], v[80:83], v[120:123]\nds_read_b64_tr_b16 v[28:29], v11 offset:38912\n"
      "v_mfma_f32_16x16x32_bf16 v[152:155], v[16:19], v[84:87], v[152:155]\nds_read_b64_tr_b16 v[30:31], v11 offset:39424\n"
      "v_mfma_f32_16x16x32_bf16 v[184:187], v[16:19], v[88:91], v[184:187]\nv_mfma_f32_16x16x32_bf16 v[216:219], v[16:19], v[92:95], v[216:219]\n"
      "s_waitcnt lgkmcnt(5)\nv_mfma_f32_16x16x32_bf16 v[220:223], v[20:23], v[92:95], v[220:223]\nds_read_b64_tr_b16 v[16:17], v6 offset:0\n"
      "v_mfma_f32_16x16x32_bf16 v[188:191], v[20:23], v[88:91], v[188:191]\nds_read_b64_tr_b16 v[18:19], v6 offset:2048\n"
      "v_mfma_f32_16x16x32_bf16 v[156:159], v[20:23], v[84:87], v[156:159]\nv_mfma_f32_16x16x32_bf16 v[124:127], v[20:23], v[80:83], v[124:127]\n"
      "s_waitcnt lgkmcnt(5)\nv_mfma_f32_16x16x32_bf16 v[224:227], v[24:27], v[80:83], v[224:227]\nds_read_b64_tr_b16 v[20:21], v7 offset:0\n"
      "v_mfma_f32_16x16x32_bf16 v[232:235], v[24:27], v[84:87], v[232:235]\nds_read_b64_tr_b16 v[22:23], v7 offset:2048\n"
      "v_mfma_f32_16x16x32_bf16 v[240:243], v[24:27], v[88:91], v[240:243]\nds_write_b32 v14, v15\n"
      "v_mfma_f32_16x16x32_bf16 v[248:251], v[24:27], v[92:95], v[248:251]\ns_waitcnt lgkmcnt(5)\n"
      "v_mfma_f32_16x16x32_bf16 v[252:255], v[28:31], v[92:95], v[252:255]\nds_read_b64_tr_b16 v[24:25], v8 offset:0\n"
      "v_mfma_f32_16x16x32_bf16 v[244:247], v[28:31], v[88:91], v[244:247]\nds_read_b64_tr_b16 v[26:27], v8 offset:2048\n"
      "v_mfma_f32_16x16x32_bf16 v[236:239], v[28:31], v[84:87], v[236:239]\nv_mfma_f32_16x16x32_bf16 v[228:231], v[28:31], v[80:83], v[228:231]\n"
      "s_branch Lend_%=\nLz0_%=:\ns_sleep 1\nds_read_b64 v[4:5], v13\ns_waitcnt lgkmcnt(0)\nv_cmp_gt_u32 vcc, %[na], v5\ns_cbranch_vccnz Lz0_%=\n"
      "v_cmp_gt_u32 vcc, %[nw], v4\ns_cbranch_vccnz Lz0_%=\ns_branch Lb0_%=\nLz2_%=:\ns_sleep 1\nds_read_b64 v[4:5], v13\ns_waitcnt lgkmcnt(0)\n"
      "v_cmp_gt_u32 vcc, %[na], v5\ns_cbranch_vccnz Lz2_%=\nv_cmp_gt_u32 vcc, %[nw], v4\ns_cbranch_vccnz Lz2_%=\ns_branch Lloop_%=\nLend_%=:\ns_nop 7\n"
      "s_waitcnt lgkmcnt(0)\nv_max3_i32 v4, v96, v97, 0\nv_max3_i32 v5, v98, v99, 0\nv_max3_i32 v6, v100, v101, 0\nv_max3_i32 v7, v102, v103, 0\n"
      "v_max3_i32 v4, v104, v105, v4\nv_max3_i32 v5, v106, v107, v5\nv_max3_i32 v6, v108, v109, v6\nv_max3_i32 v7, v110, v111, v7\n"
      "v_max3_i32 v4, v112, v113, v4\nv_max3_i32 v5, v114, v115, v5\nv_max3_i32 v6, v116, v117, v6\nv_max3_i32 v7, v118, v119, v7\n"
      "v_max3_i32 v4, v120, v121, v4\nv_max3_i32 v5, v122, v123, v5\nv_max3_i32 v6, v124, v125, v6\nv_max3_i32 v7, v126, v127, v7\n"
      "v_max3_i32 v4, v128, v129, v4\nv_max3_i32 v5, v130, v131, v5\nv_max3_i32 v6, v132, v133, v6\nv_max3_i32 v7, v134, v135, v7\n"
      "v_max3_i32 v4, v136, v137, v4\nv_max3_i32 v5, v138, v139, v5\nv_max3_i32 v6, v140, v141, v6\nv_max3_i32 v7, v142, v143, v7\n"
      "v_max3_i32 v4, v144, v145, v4\nv_max3_i32 v5, v146, v147, v5\nv_max3_i32 v6, v148, v149, v6\nv_max3_i32 v7, v150, v151, v7\n"
      "v_max3_i32 v4, v152, v153, v4\nv_max3_i32 v5, v154, v155, v5\nv_max3_i32 v6, v156, v157, v6\nv_max3_i32 v7, v158, v159, v7\n"
      "v_max3_i32 v4, v160, v161, v4\nv_max3_i32 v5, v162, v163, v5\nv_max3_i32 v6, v164, v165, v6\nv_max3_i32 v7, v166, v167, v7\n"
      "v_max3_i32 v4, v168, v169, v4\nv_max3_i32 v5, v170, v171, v5\nv_max3_i32 v6, v172, v173, v6\nv_max3_i32 v7, v174, v175, v7\n"
      "v_max3_i32 v4, v176, v177, v4\nv_max3_i32 v5, v178, v179, v5\nv_max3_i32 v6, v180, v181, v6\nv_max3_i32 v7, v182, v183, v7\n"
      "v_max3_i32 v4, v184, v185, v4\nv_max3_i32 v5, v186, v187, v5\nv_max3_i32 v6, v188, v189, v6\nv_max3_i32 v7, v190, v191, v7\n"
      "v_max3_i32 v4, v192, v193, v4\nv_max3_i32 v5, v194, v195, v5\nv_max3_i32 v6, v196, v197, v6\nv_max3_i32 v7, v198, v199, v7\n"
      "v_max3_i32 v4, v200, v201, v4\nv_max3_i32 v5, v202, v203, v5\nv_max3_i32 v6, v204, v205, v6\nv_max3_i32 v7, v206, v207, v7\n"
      "v_max3_i32 v4, v208, v209, v4\nv_max3_i32 v5, v210, v211, v5\nv_max3_i32 v6, v212, v213, v6\nv_max3_i32 v7, v214, v215, v7\n"
      "v_max3_i32 v4, v216, v217, v4\nv_max3_i32 v5, v218, v219, v5\nv_max3_i32 v6, v220, v221, v6\nv_max3_i32 v7, v222, v223, v7\n"
      "v_max3_i32 v4, v224, v225, v4\nv_max3_i32 v5, v226, v227, v5\nv_max3_i32 v6, v228, v229, v6\nv_max3_i32 v7, v230, v231, v7\n"
      "v_max3_i32 v4, v232, v233, v4\nv_max3_i32 v5, v234, v235, v5\nv_max3_i32 v6, v236, v237, v6\nv_max3_i32 v7, v238, v239, v7\n"
      "v_max3_i32 v4, v240, v241, v4\nv_max3_i32 v5, v242, v243, v5\nv_max3_i32 v6, v244, v245, v6\nv_max3_i32 v7, v246, v247, v7\n"
      "v_max3_i32 v4, v248, v249, v4\nv_max3_i32 v5, v250, v251, v5\nv_max3_i32 v6, v252, v253, v6\nv_max3_i32 v7, v254, v255, v7\n"
      "v_max3_i32 v15, v4, v5, v6\nv_max_i32 v15, v15, v7\ns_waitcnt vmcnt(0)\n"
      : "={v[96:127]}"(a0), "={v[128:159]}"(a1), "={v[160:191]}"(a2), "={v[192:223]}"(a3), "={v[224:255]}"(a4),
        "+{v6}"(w0), "+{v7}"(w1), "+{v8}"(w2), "+{v9}"(w3), "+{v10}"(w4), "+{v11}"(w5), "+{v15}"(fv),
        [xo] "+s"(xo), [na] "+s"(na), [nw] "+s"(nw), [cnt] "+s"(cnt), [dl] "+s"(dl), [sa] "=&s"(sa), [sw] "=&s"(sw)
      : "{v12}"(l16), "{v13}"(ctla), "{v14}"(fa), [rs] "s"(ra), [cst] "s"(cst)
      : "memory", "scc", "vcc",
        "v4", "v5", "v16", "v17", "v18", "v19", "v20", "v21", "v22", "v23", "v24", "v25", "v26", "v27", "v28", "v29",
        "v30", "v31", "v32", "v33", "v34", "v35", "v36", "v37", "v38", "v39", "v40", "v41", "v42", "v43", "v44", "v45",
        "v46", "v47", "v48", "v49", "v50", "v51", "v52", "v53", "v54", "v55", "v56", "v57", "v58", "v59", "v60", "v61",
        "v62", "v63", "v64", "v65", "v66", "v67", "v68", "v69", "v70", "v71", "v72", "v73", "v74", "v75", "v76", "v77",
        "v78", "v79", "v80", "v81", "v82", "v83", "v84", "v85", "v86", "v87", "v88", "v89", "v90", "v91", "v92", "v93",
        "v94", "v95");
  const bool real = na != 0;
  int ln;
  asm volatile("v_mbcnt_lo_u32_b32 %0, -1, 0\n\tv_mbcnt_hi_u32_b32 %0, -1, %0" : "=v"(ln));
  const unsigned wm = wave_max(fv);
  unsigned tick = 0;
  if (ln == 0 && real) {
    __hip_atomic_fetch_max((LAS unsigned*)(lds + CT_MAX), wm, __ATOMIC_RELAXED, __HIP_MEMORY_SCOPE_WORKGROUP);
    asm volatile("" ::: "memory");
    tick = lds_inc(lds + CT_MAXN);
  }
  if (rfl(tick) == 3) {
    const unsigned gm = lds_ld(lds + CT_MAX);
    if (ln == 0) st_granule(ws + WS_AMX + b * 128, call, gm);
    lds_st(lds + CT_MAXSENT, 1);
  }
  const int fr = ln & 15, fq = ln >> 4;
  const i32x4 ro = mkr(outp, real ? 64 * NL : 0);
  const unsigned lb = real ? wv * 10240 : CT_DUM;
  const unsigned lw = lb + fr * 128 + ((fr & 7) << 4) + fq * 4, lw2 = lb + 8192 + fr * 32 + fq * 4;
  const unsigned lr = lb + (ln >> 3) * 128 + (((ln ^ (ln >> 3)) & 7) << 4), lr2 = lb + 8192 + ln * 16;
  const unsigned vo = (ln >> 3) * NL + c128 + (ln & 7) * 16, vo2 = cb == 12 ? 0x80000000u : (ln >> 1) * NL + c32 + (ln & 1) * 16;
  if (real) while (rfl(lds_ld(lds + CT_MAXSENT)) == 0) __builtin_amdgcn_s_sleep(0);
  const unsigned mine = rfl(lds_ld(lds + CT_MAX));
  unsigned done = ~0u, state;
  do {
    const u64 v = __hip_atomic_load((LAS u64*)(lds + CT_AMAX), __ATOMIC_RELAXED, __HIP_MEMORY_SCOPE_WORKGROUP);
    const unsigned cur = umax(rfl((unsigned)v), mine);
    state = real ? rfl((unsigned)(v >> 32)) : 2;
    if (state == 0 || cur == done) { __builtin_amdgcn_s_sleep(1); continue; }
    if (done != ~0u) asm volatile("s_waitcnt vmcnt(0)" ::: "memory");
    done = cur;
    const float inv = 1.0f / (__uint_as_float(cur) / 127.0f);
    unsigned so;
    asm volatile(
        "s_waitcnt lgkmcnt(0)\nv_mov_b32 v16, %[inv]\nv_mov_b32 v17, %[inv]\nv_xor_b32 v75, 16, %[lw]\nv_xor_b32 v76, 32, %[lw]\nv_xor_b32 v77, 48, %[lw]\n"
        "v_xor_b32 v78, 64, %[lw]\nv_xor_b32 v79, 80, %[lw]\nv_xor_b32 v80, 96, %[lw]\nv_xor_b32 v81, 112, %[lw]\nv_mov_b32 v74, %[lw]\n"
        "v_pk_mul_f32 v[18:19], v[224:225], v[16:17]\nv_pk_mul_f32 v[20:21], v[226:227], v[16:17]\nv_pk_mul_f32 v[22:23], v[228:229], v[16:17]\n"
        "v_pk_mul_f32 v[24:25], v[230:231], v[16:17]\nv_pk_mul_f32 v[26:27], v[232:233], v[16:17]\nv_pk_mul_f32 v[28:29], v[234:235], v[16:17]\n"
        "v_pk_mul_f32 v[30:31], v[236:237], v[16:17]\nv_pk_mul_f32 v[32:33], v[238:239], v[16:17]\nv_cvt_pk_u8_f32 v50, v18, 0, 0\n"
        "v_cvt_pk_u8_f32 v51, v22, 0, 0\nv_pk_mul_f32 v[34:35], v[240:241], v[16:17]\nv_cvt_pk_u8_f32 v52, v26, 0, 0\nv_cvt_pk_u8_f32 v53, v30, 0, 0\n"
        "v_pk_mul_f32 v[36:37], v[242:243], v[16:17]\nv_cvt_pk_u8_f32 v50, v19, 1, v50\nv_cvt_pk_u8_f32 v51, v23, 1, v51\n"
        "v_pk_mul_f32 v[38:39], v[244:245], v[16:17]\nv_cvt_pk_u8_f32 v52, v27, 1, v52\nv_cvt_pk_u8_f32 v53, v31, 1, v53\n"
        "v_pk_mul_f32 v[40:41], v[246:247], v[16:17]\nv_cvt_pk_u8_f32 v50, v20, 2, v50\nv_cvt_pk_u8_f32 v51, v24, 2, v51\n"
        "v_pk_mul_f32 v[42:43], v[248:249], v[16:17]\nv_cvt_pk_u8_f32 v52, v28, 2, v52\nv_cvt_pk_u8_f32 v53, v32, 2, v53\n"
        "v_pk_mul_f32 v[44:45], v[250:251], v[16:17]\nv_cvt_pk_u8_f32 v50, v21, 3, v50\nv_cvt_pk_u8_f32 v51, v25, 3, v51\n"
        "v_pk_mul_f32 v[46:47], v[252:253], v[16:17]\nv_cvt_pk_u8_f32 v52, v29, 3, v52\nv_cvt_pk_u8_f32 v53, v33, 3, v53\n"
        "v_pk_mul_f32 v[48:49], v[254:255], v[16:17]\nds_write_b32 %[lw2], v50 offset:0\nds_write_b32 %[lw2], v51 offset:16\n"
        "ds_write_b32 %[lw2], v52 offset:512\nds_write_b32 %[lw2], v53 offset:528\nv_cvt_pk_u8_f32 v54, v34, 0, 0\nv_cvt_pk_u8_f32 v55, v38, 0, 0\n"
        "v_pk_mul_f32 v[18:19], v[96:97], v[16:17]\nv_cvt_pk_u8_f32 v56, v42, 0, 0\nv_cvt_pk_u8_f32 v57, v46, 0, 0\nv_pk_mul_f32 v[20:21], v[98:99], v[16:17]\n"
        "v_cvt_pk_u8_f32 v54, v35, 1, v54\nv_cvt_pk_u8_f32 v55, v39, 1, v55\nv_pk_mul_f32 v[22:23], v[100:101], v[16:17]\nv_cvt_pk_u8_f32 v56, v43, 1, v56\n"
        "v_cvt_pk_u8_f32 v57, v47, 1, v57\nv_pk_mul_f32 v[24:25], v[102:103], v[16:17]\nv_cvt_pk_u8_f32 v54, v36, 2, v54\nv_cvt_pk_u8_f32 v55, v40, 2, v55\n"
        "v_pk_mul_f32 v[26:27], v[104:105], v[16:17]\nv_cvt_pk_u8_f32 v56, v44, 2, v56\nv_cvt_pk_u8_f32 v57, v48, 2, v57\n"
        "v_pk_mul_f32 v[28:29], v[106:107], v[16:17]\nv_cvt_pk_u8_f32 v54, v37, 3, v54\nv_cvt_pk_u8_f32 v55, v41, 3, v55\n"
        "v_pk_mul_f32 v[30:31], v[108:109], v[16:17]\nv_cvt_pk_u8_f32 v56, v45, 3, v56\nv_cvt_pk_u8_f32 v57, v49, 3, v57\n"
        "v_pk_mul_f32 v[32:33], v[110:111], v[16:17]\nds_write_b32 %[lw2], v54 offset:1024\nds_write_b32 %[lw2], v55 offset:1040\n"
        "ds_write_b32 %[lw2], v56 offset:1536\nds_write_b32 %[lw2], v57 offset:1552\nds_read_b128 v[58:61], %[lr2] offset:0\n"
        "ds_read_b128 v[62:65], %[lr2] offset:1024\nv_cvt_pk_u8_f32 v50, v18, 0, 0\nv_cvt_pk_u8_f32 v51, v22, 0, 0\n"
        "v_pk_mul_f32 v[34:35], v[112:113], v[16:17]\nv_cvt_pk_u8_f32 v52, v26, 0, 0\nv_cvt_pk_u8_f32 v53, v30, 0, 0\n"
        "v_pk_mul_f32 v[36:37], v[114:115], v[16:17]\nv_cvt_pk_u8_f32 v50, v19, 1, v50\nv_cvt_pk_u8_f32 v51, v23, 1, v51\n"
        "v_pk_mul_f32 v[38:39], v[116:117], v[16:17]\nv_cvt_pk_u8_f32 v52, v27, 1, v52\nv_cvt_pk_u8_f32 v53, v31, 1, v53\n"
        "v_pk_mul_f32 v[40:41], v[118:119], v[16:17]\nv_cvt_pk_u8_f32 v50, v20, 2, v50\nv_cvt_pk_u8_f32 v51, v24, 2, v51\n"
        "v_pk_mul_f32 v[42:43], v[120:121], v[16:17]\nv_cvt_pk_u8_f32 v52, v28, 2, v52\nv_cvt_pk_u8_f32 v53, v32, 2, v53\n"
        "v_pk_mul_f32 v[44:45], v[122:123], v[16:17]\nv_cvt_pk_u8_f32 v50, v21, 3, v50\nv_cvt_pk_u8_f32 v51, v25, 3, v51\n"
        "v_pk_mul_f32 v[46:47], v[124:125], v[16:17]\nv_cvt_pk_u8_f32 v52, v29, 3, v52\nv_cvt_pk_u8_f32 v53, v33, 3, v53\n"
        "v_pk_mul_f32 v[48:49], v[126:127], v[16:17]\nds_write_b32 v74, v50 offset:0\nds_write_b32 v75, v51 offset:0\nds_write_b32 v76, v52 offset:0\n"
        "ds_write_b32 v77, v53 offset:0\ns_waitcnt lgkmcnt(4)\ns_mov_b32 %[so], 0\nbuffer_store_dwordx4 v[58:61], %[vo2], %[ro], %[so] offen\n"
        "s_mov_b32 %[so], 65536\nbuffer_store_dwordx4 v[62:65], %[vo2], %[ro], %[so] offen\nv_cvt_pk_u8_f32 v54, v34, 0, 0\nv_cvt_pk_u8_f32 v55, v38, 0, 0\n"
        "v_pk_mul_f32 v[18:19], v[128:129], v[16:17]\nv_cvt_pk_u8_f32 v56, v42, 0, 0\nv_cvt_pk_u8_f32 v57, v46, 0, 0\n"
        "v_pk_mul_f32 v[20:21], v[130:131], v[16:17]\nv_cvt_pk_u8_f32 v54, v35, 1, v54\nv_cvt_pk_u8_f32 v55, v39, 1, v55\n"
        "v_pk_mul_f32 v[22:23], v[132:133], v[16:17]\nv_cvt_pk_u8_f32 v56, v43, 1, v56\nv_cvt_pk_u8_f32 v57, v47, 1, v57\n"
        "v_pk_mul_f32 v[24:25], v[134:135], v[16:17]\nv_cvt_pk_u8_f32 v54, v36, 2, v54\nv_cvt_pk_u8_f32 v55, v40, 2, v55\n"
        "v_pk_mul_f32 v[26:27], v[136:137], v[16:17]\nv_cvt_pk_u8_f32 v56, v44, 2, v56\nv_cvt_pk_u8_f32 v57, v48, 2, v57\n"
        "v_pk_mul_f32 v[28:29], v[138:139], v[16:17]\nv_cvt_pk_u8_f32 v54, v37, 3, v54\nv_cvt_pk_u8_f32 v55, v41, 3, v55\n"
        "v_pk_mul_f32 v[30:31], v[140:141], v[16:17]\nv_cvt_pk_u8_f32 v56, v45, 3, v56\nv_cvt_pk_u8_f32 v57, v49, 3, v57\n"
        "v_pk_mul_f32 v[32:33], v[142:143], v[16:17]\nds_write_b32 v78, v54 offset:0\nds_write_b32 v79, v55 offset:0\nds_write_b32 v80, v56 offset:0\n"
        "ds_write_b32 v81, v57 offset:0\nds_read_b128 v[66:69], %[lr] offset:0\nds_read_b128 v[70:73], %[lr] offset:1024\nv_cvt_pk_u8_f32 v50, v18, 0, 0\n"
        "v_cvt_pk_u8_f32 v51, v22, 0, 0\nv_pk_mul_f32 v[34:35], v[144:145], v[16:17]\nv_cvt_pk_u8_f32 v52, v26, 0, 0\nv_cvt_pk_u8_f32 v53, v30, 0, 0\n"
        "v_pk_mul_f32 v[36:37], v[146:147], v[16:17]\nv_cvt_pk_u8_f32 v50, v19, 1, v50\nv_cvt_pk_u8_f32 v51, v23, 1, v51\n"
        "v_pk_mul_f32 v[38:39], v[148:149], v[16:17]\nv_cvt_pk_u8_f32 v52, v27, 1, v52\nv_cvt_pk_u8_f32 v53, v31, 1, v53\n"
        "v_pk_mul_f32 v[40:41], v[150:151], v[16:17]\nv_cvt_pk_u8_f32 v50, v20, 2, v50\nv_cvt_pk_u8_f32 v51, v24, 2, v51\n"
        "v_pk_mul_f32 v[42:43], v[152:153], v[16:17]\nv_cvt_pk_u8_f32 v52, v28, 2, v52\nv_cvt_pk_u8_f32 v53, v32, 2, v53\n"
        "v_pk_mul_f32 v[44:45], v[154:155], v[16:17]\nv_cvt_pk_u8_f32 v50, v21, 3, v50\nv_cvt_pk_u8_f32 v51, v25, 3, v51\n"
        "v_pk_mul_f32 v[46:47], v[156:157], v[16:17]\nv_cvt_pk_u8_f32 v52, v29, 3, v52\nv_cvt_pk_u8_f32 v53, v33, 3, v53\n"
        "v_pk_mul_f32 v[48:49], v[158:159], v[16:17]\nds_write_b32 v74, v50 offset:2048\nds_write_b32 v75, v51 offset:2048\nds_write_b32 v76, v52 offset:2048\n"
        "ds_write_b32 v77, v53 offset:2048\ns_waitcnt lgkmcnt(4)\ns_mov_b32 %[so], 0\nbuffer_store_dwordx4 v[66:69], %[vo], %[ro], %[so] offen sc0 sc1\n"
        "s_mov_b32 %[so], 16384\nbuffer_store_dwordx4 v[70:73], %[vo], %[ro], %[so] offen sc0 sc1\nv_cvt_pk_u8_f32 v54, v34, 0, 0\n"
        "v_cvt_pk_u8_f32 v55, v38, 0, 0\nv_pk_mul_f32 v[18:19], v[160:161], v[16:17]\nv_cvt_pk_u8_f32 v56, v42, 0, 0\nv_cvt_pk_u8_f32 v57, v46, 0, 0\n"
        "v_pk_mul_f32 v[20:21], v[162:163], v[16:17]\nv_cvt_pk_u8_f32 v54, v35, 1, v54\nv_cvt_pk_u8_f32 v55, v39, 1, v55\n"
        "v_pk_mul_f32 v[22:23], v[164:165], v[16:17]\nv_cvt_pk_u8_f32 v56, v43, 1, v56\nv_cvt_pk_u8_f32 v57, v47, 1, v57\n"
        "v_pk_mul_f32 v[24:25], v[166:167], v[16:17]\nv_cvt_pk_u8_f32 v54, v36, 2, v54\nv_cvt_pk_u8_f32 v55, v40, 2, v55\n"
        "v_pk_mul_f32 v[26:27], v[168:169], v[16:17]\nv_cvt_pk_u8_f32 v56, v44, 2, v56\nv_cvt_pk_u8_f32 v57, v48, 2, v57\n"
        "v_pk_mul_f32 v[28:29], v[170:171], v[16:17]\nv_cvt_pk_u8_f32 v54, v37, 3, v54\nv_cvt_pk_u8_f32 v55, v41, 3, v55\n"
        "v_pk_mul_f32 v[30:31], v[172:173], v[16:17]\nv_cvt_pk_u8_f32 v56, v45, 3, v56\nv_cvt_pk_u8_f32 v57, v49, 3, v57\n"
        "v_pk_mul_f32 v[32:33], v[174:175], v[16:17]\nds_write_b32 v78, v54 offset:2048\nds_write_b32 v79, v55 offset:2048\nds_write_b32 v80, v56 offset:2048\n"
        "ds_write_b32 v81, v57 offset:2048\nds_read_b128 v[58:61], %[lr] offset:2048\nds_read_b128 v[62:65], %[lr] offset:3072\n"
        "v_cvt_pk_u8_f32 v50, v18, 0, 0\nv_cvt_pk_u8_f32 v51, v22, 0, 0\nv_pk_mul_f32 v[34:35], v[176:177], v[16:17]\nv_cvt_pk_u8_f32 v52, v26, 0, 0\n"
        "v_cvt_pk_u8_f32 v53, v30, 0, 0\nv_pk_mul_f32 v[36:37], v[178:179], v[16:17]\nv_cvt_pk_u8_f32 v50, v19, 1, v50\nv_cvt_pk_u8_f32 v51, v23, 1, v51\n"
        "v_pk_mul_f32 v[38:39], v[180:181], v[16:17]\nv_cvt_pk_u8_f32 v52, v27, 1, v52\nv_cvt_pk_u8_f32 v53, v31, 1, v53\n"
        "v_pk_mul_f32 v[40:41], v[182:183], v[16:17]\nv_cvt_pk_u8_f32 v50, v20, 2, v50\nv_cvt_pk_u8_f32 v51, v24, 2, v51\n"
        "v_pk_mul_f32 v[42:43], v[184:185], v[16:17]\nv_cvt_pk_u8_f32 v52, v28, 2, v52\nv_cvt_pk_u8_f32 v53, v32, 2, v53\n"
        "v_pk_mul_f32 v[44:45], v[186:187], v[16:17]\nv_cvt_pk_u8_f32 v50, v21, 3, v50\nv_cvt_pk_u8_f32 v51, v25, 3, v51\n"
        "v_pk_mul_f32 v[46:47], v[188:189], v[16:17]\nv_cvt_pk_u8_f32 v52, v29, 3, v52\nv_cvt_pk_u8_f32 v53, v33, 3, v53\n"
        "v_pk_mul_f32 v[48:49], v[190:191], v[16:17]\nds_write_b32 v74, v50 offset:4096\nds_write_b32 v75, v51 offset:4096\nds_write_b32 v76, v52 offset:4096\n"
        "ds_write_b32 v77, v53 offset:4096\ns_waitcnt lgkmcnt(4)\ns_mov_b32 %[so], 32768\nbuffer_store_dwordx4 v[58:61], %[vo], %[ro], %[so] offen sc0 sc1\n"
        "s_mov_b32 %[so], 49152\nbuffer_store_dwordx4 v[62:65], %[vo], %[ro], %[so] offen sc0 sc1\nv_cvt_pk_u8_f32 v54, v34, 0, 0\n"
        "v_cvt_pk_u8_f32 v55, v38, 0, 0\nv_pk_mul_f32 v[18:19], v[192:193], v[16:17]\nv_cvt_pk_u8_f32 v56, v42, 0, 0\nv_cvt_pk_u8_f32 v57, v46, 0, 0\n"
        "v_pk_mul_f32 v[20:21], v[194:195], v[16:17]\nv_cvt_pk_u8_f32 v54, v35, 1, v54\nv_cvt_pk_u8_f32 v55, v39, 1, v55\n"
        "v_pk_mul_f32 v[22:23], v[196:197], v[16:17]\nv_cvt_pk_u8_f32 v56, v43, 1, v56\nv_cvt_pk_u8_f32 v57, v47, 1, v57\n"
        "v_pk_mul_f32 v[24:25], v[198:199], v[16:17]\nv_cvt_pk_u8_f32 v54, v36, 2, v54\nv_cvt_pk_u8_f32 v55, v40, 2, v55\n"
        "v_pk_mul_f32 v[26:27], v[200:201], v[16:17]\nv_cvt_pk_u8_f32 v56, v44, 2, v56\nv_cvt_pk_u8_f32 v57, v48, 2, v57\n"
        "v_pk_mul_f32 v[28:29], v[202:203], v[16:17]\nv_cvt_pk_u8_f32 v54, v37, 3, v54\nv_cvt_pk_u8_f32 v55, v41, 3, v55\n"
        "v_pk_mul_f32 v[30:31], v[204:205], v[16:17]\nv_cvt_pk_u8_f32 v56, v45, 3, v56\nv_cvt_pk_u8_f32 v57, v49, 3, v57\n"
        "v_pk_mul_f32 v[32:33], v[206:207], v[16:17]\nds_write_b32 v78, v54 offset:4096\nds_write_b32 v79, v55 offset:4096\nds_write_b32 v80, v56 offset:4096\n"
        "ds_write_b32 v81, v57 offset:4096\nds_read_b128 v[66:69], %[lr] offset:4096\nds_read_b128 v[70:73], %[lr] offset:5120\n"
        "v_cvt_pk_u8_f32 v50, v18, 0, 0\nv_cvt_pk_u8_f32 v51, v22, 0, 0\nv_pk_mul_f32 v[34:35], v[208:209], v[16:17]\nv_cvt_pk_u8_f32 v52, v26, 0, 0\n"
        "v_cvt_pk_u8_f32 v53, v30, 0, 0\nv_pk_mul_f32 v[36:37], v[210:211], v[16:17]\nv_cvt_pk_u8_f32 v50, v19, 1, v50\nv_cvt_pk_u8_f32 v51, v23, 1, v51\n"
        "v_pk_mul_f32 v[38:39], v[212:213], v[16:17]\nv_cvt_pk_u8_f32 v52, v27, 1, v52\nv_cvt_pk_u8_f32 v53, v31, 1, v53\n"
        "v_pk_mul_f32 v[40:41], v[214:215], v[16:17]\nv_cvt_pk_u8_f32 v50, v20, 2, v50\nv_cvt_pk_u8_f32 v51, v24, 2, v51\n"
        "v_pk_mul_f32 v[42:43], v[216:217], v[16:17]\nv_cvt_pk_u8_f32 v52, v28, 2, v52\nv_cvt_pk_u8_f32 v53, v32, 2, v53\n"
        "v_pk_mul_f32 v[44:45], v[218:219], v[16:17]\nv_cvt_pk_u8_f32 v50, v21, 3, v50\nv_cvt_pk_u8_f32 v51, v25, 3, v51\n"
        "v_pk_mul_f32 v[46:47], v[220:221], v[16:17]\nv_cvt_pk_u8_f32 v52, v29, 3, v52\nv_cvt_pk_u8_f32 v53, v33, 3, v53\n"
        "v_pk_mul_f32 v[48:49], v[222:223], v[16:17]\nds_write_b32 v74, v50 offset:6144\nds_write_b32 v75, v51 offset:6144\nds_write_b32 v76, v52 offset:6144\n"
        "ds_write_b32 v77, v53 offset:6144\ns_waitcnt lgkmcnt(4)\ns_mov_b32 %[so], 65536\nbuffer_store_dwordx4 v[66:69], %[vo], %[ro], %[so] offen sc0 sc1\n"
        "s_mov_b32 %[so], 81920\nbuffer_store_dwordx4 v[70:73], %[vo], %[ro], %[so] offen sc0 sc1\nv_cvt_pk_u8_f32 v54, v34, 0, 0\n"
        "v_cvt_pk_u8_f32 v55, v38, 0, 0\nv_cvt_pk_u8_f32 v56, v42, 0, 0\nv_cvt_pk_u8_f32 v57, v46, 0, 0\nv_cvt_pk_u8_f32 v54, v35, 1, v54\n"
        "v_cvt_pk_u8_f32 v55, v39, 1, v55\nv_cvt_pk_u8_f32 v56, v43, 1, v56\nv_cvt_pk_u8_f32 v57, v47, 1, v57\nv_cvt_pk_u8_f32 v54, v36, 2, v54\n"
        "v_cvt_pk_u8_f32 v55, v40, 2, v55\nv_cvt_pk_u8_f32 v56, v44, 2, v56\nv_cvt_pk_u8_f32 v57, v48, 2, v57\nv_cvt_pk_u8_f32 v54, v37, 3, v54\n"
        "v_cvt_pk_u8_f32 v55, v41, 3, v55\nv_cvt_pk_u8_f32 v56, v45, 3, v56\nv_cvt_pk_u8_f32 v57, v49, 3, v57\nds_write_b32 v78, v54 offset:6144\n"
        "ds_write_b32 v79, v55 offset:6144\nds_write_b32 v80, v56 offset:6144\nds_write_b32 v81, v57 offset:6144\nds_read_b128 v[58:61], %[lr] offset:6144\n"
        "ds_read_b128 v[62:65], %[lr] offset:7168\ns_waitcnt lgkmcnt(0)\ns_mov_b32 %[so], 98304\n"
        "buffer_store_dwordx4 v[58:61], %[vo], %[ro], %[so] offen sc0 sc1\ns_mov_b32 %[so], 114688\n"
        "buffer_store_dwordx4 v[62:65], %[vo], %[ro], %[so] offen sc0 sc1\ns_nop 1\n"
        : [so] "=&s"(so)
        : "{v[96:127]}"(a0), "{v[128:159]}"(a1), "{v[160:191]}"(a2), "{v[192:223]}"(a3), "{v[224:255]}"(a4),
          [inv] "v"(inv), [lw] "v"(lw), [lw2] "v"(lw2), [lr] "v"(lr), [lr2] "v"(lr2), [vo] "v"(vo), [vo2] "v"(vo2), [ro] "s"(ro)
        : "memory", "v16", "v17", "v18", "v19", "v20", "v21", "v22", "v23", "v24", "v25", "v26", "v27", "v28", "v29", "v30", "v31", "v32", "v33", "v34", "v35", "v36", "v37", "v38", "v39", "v40", "v41", "v42", "v43", "v44", "v45", "v46", "v47", "v48", "v49", "v50", "v51", "v52", "v53", "v54", "v55", "v56", "v57", "v58", "v59", "v60", "v61", "v62", "v63", "v64", "v65", "v66", "v67", "v68", "v69", "v70", "v71", "v72", "v73", "v74", "v75", "v76", "v77", "v78", "v79", "v80", "v81");
  } while (state < 2);
  if (state == 3) {
    const u32x4 o = {0x80808080u, 0x80808080u, 0x80808080u, 0x80808080u};
    asm volatile("s_waitcnt vmcnt(0)" ::: "memory");
#pragma unroll
    for (int k = 0; k < 10; ++k)
      asm volatile("buffer_store_dwordx4 %0, %1, %2, %3 offen\n\ts_nop 1" :: "v"(o), "v"(k < 8 ? vo : vo2), "s"(ro), "s"(k < 8 ? k * 8 * NL : (k - 8) * 32 * NL) : "memory");
  }
}
}

extern "C" void kernel_launch(int rank, int nranks, ncclComm_t comm,
                              hipStream_t stream,
                              void* const* d_in, const int* in_sizes, int n_in,
                              void* d_out, int out_size,
                              const int* in_dtypes, int out_dtype,
                              void* const* d_out_all,
                              void* const* d_ws_all, size_t ws_bytes) {
  using namespace agq;
  if (nranks != NR || (unsigned)rank >= (unsigned)NR || n_in != 2 || in_sizes[0] != MS * KK || in_sizes[1] != KK * NL ||
      out_size != MT * NL || ws_bytes < WS_END) { fprintf(stderr, "unexpected problem shape\n"); return; }
  static bool once = false;
  if (!once) {
    HIP_CHECK(hipFuncSetAttribute((const void*)ag_gemm_q8, hipFuncAttributeMaxDynamicSharedMemorySize, LDS_B));
    once = true;
  }
  Params P;
  P.x = (const char*)d_in[0]; P.w = (const char*)d_in[1]; P.out = (char*)d_out;
  for (int p = 0; p < NR; ++p) { P.ws[p] = (char*)d_ws_all[p]; P.rot[p] = (char*)d_ws_all[(rank + 1 + p) % NR]; }
  P.self = P.ws[rank];
  P.rank = rank; P.pad = 0;
  hipLaunchKernelGGL(ag_gemm_q8, dim3(NWG), dim3(512), LDS_B, stream, P);
  HIP_CHECK(hipGetLastError());
}
```
